```python
import numpy as np

kernel_name = "gpu_gemm_a2a_m4096_k4096_n16384_w8a8_dqrelu"


ABSMAX_RTOL = 0.001

COMPUTE_DTYPE = "int8"
EPILOGUE = "dequant_relu"
IN_QUANT = "int8"
OUT_QUANT = None
IN_DTYPES = {"x": "int8", "w_mat": "int8",
             "scale_x": "float32", "scale_w": "float32"}
WORLD_SIZE = 8

def setup_inputs():

    x = np.random.randint(-127, 128, size=(4096, 4096)).astype(np.int8)
    w_mat = np.random.randint(-127, 128, size=(4096, 16384)).astype(np.int8)
    scale_x = (0.02 * (1.0 + np.random.rand(1))).astype(np.float32)
    scale_w = (0.02 * (1.0 + np.random.rand(1))).astype(np.float32)
    return {"x": x, "w_mat": w_mat, "scale_x": scale_x, "scale_w": scale_w}

def reference(x, w_mat, scale_x, scale_w):
    acc = x.astype(np.float64) @ w_mat.astype(np.float64)
    y = (acc.astype(np.float32) * (scale_x * scale_w)).astype(np.float32)
    return np.maximum(y, 0.0).astype(np.float32)
```

```cpp
#include <hip/hip_runtime.h>
#include <rccl/rccl.h>
#include <cstdio>

#define HIP_CHECK(expr) do { hipError_t _e = (expr); if (_e != hipSuccess) { \
    fprintf(stderr, "HIP error %s at %s:%d\n", hipGetErrorString(_e), __FILE__, __LINE__); \
    return; } } while (0)

namespace {

#define LAS __attribute__((address_space(3)))
#define GAS __attribute__((address_space(1)))
typedef unsigned u32x4 __attribute__((ext_vector_type(4)));
typedef int i32x4 __attribute__((ext_vector_type(4)));
typedef int i32x2 __attribute__((ext_vector_type(2)));
typedef float f32x4 __attribute__((ext_vector_type(4)));

constexpr int NR = 8;
constexpr int KD = 4096, ND = 16384;
constexpr int MS = 512, NS = 2048;
constexpr int GRID = 256, BLOCK = 512;
constexpr int IMG = 16384;
constexpr int STG = 3 * IMG;
constexpr int KSTEPS = KD / 128;
constexpr int D = 3;
constexpr int LDS_BYTES = D * STG + 16;

constexpr size_t WS_CNT = 0;
constexpr size_t WS_TMO = 4096;
constexpr size_t WS_FLAG = 8192;
constexpr size_t WS_X = 1u << 20;
constexpr size_t XBYTES = (size_t)MS * KD;
constexpr size_t WS_NEED = WS_X + 2 * XBYTES;
constexpr unsigned long long DEADLINE_TICKS = 50000000ull;

struct Args {
    const signed char* x; const signed char* w; const float* sx; const float* sw;
    float* out; char* ws_all[NR]; int rank; int pad;
};

template <int N> __device__ __forceinline__ void wait_vm() { asm volatile("s_waitcnt vmcnt(%0)" :: "n"(N) : "memory"); }
#define STAMP(i) do {} while (0)
#define INV_POS 2
#define STORE_MODE 2
__device__ __forceinline__ void store_out(float* p, f32x4 y) {
    asm volatile("global_store_dwordx4 %0, %1, off sc1\n\ts_nop 1" :: "v"(p), "v"(y) : "memory");
}

__global__ __launch_bounds__(BLOCK, 2) void gemm_gather_kernel(Args a) {
    __shared__ __attribute__((aligned(16))) char smem[LDS_BYTES];
    LAS char* lds = (LAS char*)smem;
    const int tid = threadIdx.x, lane = tid & 63, wid = __builtin_amdgcn_readfirstlane(tid >> 6);
    const int b = blockIdx.x, rank = a.rank;
    char* myws = a.ws_all[rank];
    char* lanews = a.ws_all[lane & (NR - 1)];
    STAMP(0);

    unsigned call;
    float scale = a.sx[0] * a.sw[0];
    {
        const int prg = b >> 6, pks = (b >> 1) & 31, p = (b & 1) * 8192 + tid * 16;
        const int row = p >> 7, chunk = ((p >> 4) & 7) ^ ((row >> 1) & 7);
        const u32x4 v = *(const u32x4*)(a.x + (size_t)(prg * 128 + row) * KD + pks * 128 + chunk * 16);
        call = __hip_atomic_load((unsigned*)(myws + WS_CNT) + b, __ATOMIC_RELAXED, __HIP_MEMORY_SCOPE_AGENT) + 1u;
        call = __builtin_amdgcn_readfirstlane(call);
        asm volatile("" : "+v"(scale));
        char* dst = myws + WS_X + (size_t)(call & 1u) * XBYTES + (size_t)(prg * KSTEPS + pks) * IMG + p;
        asm volatile("global_store_dwordx4 %0, %1, off sc0 sc1\n\ts_nop 1" :: "v"(dst), "v"(v) : "memory");
    }
    const int grp = b & 7, idx = b >> 3;
    const int peer = (rank + grp) & 7;
    const int rg = idx & 3, cg = idx >> 2;
    const char* peerws = a.ws_all[peer];
    asm volatile("s_waitcnt vmcnt(0)" ::: "memory");
    __builtin_amdgcn_s_barrier();
    STAMP(1);
    if (wid == 0) {
        if (lane < NR) __hip_atomic_store((GAS unsigned*)(lanews + WS_FLAG) + rank * GRID + b, call, __ATOMIC_RELAXED, __HIP_MEMORY_SCOPE_SYSTEM);
        if (lane == 0) __hip_atomic_store((unsigned*)(myws + WS_CNT) + b, call, __ATOMIC_RELAXED, __HIP_MEMORY_SCOPE_AGENT);
    }

    const char* gA = peerws + WS_X + (size_t)(call & 1u) * XBYTES + (size_t)rg * KSTEPS * IMG + wid * 2048;
    const char* gB = (const char*)a.w + (size_t)rank * NS + cg * 256;
    const unsigned ldsA = (unsigned)(size_t)lds + wid * 2048, ldsB = (unsigned)(size_t)lds + IMG + wid * 4096;
    const unsigned voffA = lane * 16;
    unsigned voff[4];
#pragma unroll
    for (int i = 0; i < 4; ++i) { const int k = wid * 16 + i * 4 + (lane >> 4);
        const int c = (lane & 15) ^ ((k & 7) | (((k >> 4) & 1) << 3));
        voff[i] = k * ND + c * 16 - i * 1024; }
    auto stageA = [&](int s, unsigned slot) {
        const char* g = gA + (size_t)s * IMG;
        const unsigned l = ldsA + slot;
        unsigned keep;
        asm volatile("s_mov_b32 %0, m0\n\ts_mov_b32 m0, %3\n\ts_nop 0\n\t"
                     "global_load_lds_dwordx4 %1, %2\n\t"
                     "global_load_lds_dwordx4 %1, %2 offset:1024\n\t"
                     "s_mov_b32 m0, %0"
                     : "=&s"(keep) : "v"(voffA), "s"(g), "s"(l) : "memory");
    };
    auto stageB = [&](int s, unsigned slot) {
        const char* g = gB + (size_t)s * 128 * ND;
        const unsigned l = ldsB + slot;
        unsigned keep;
        asm volatile("s_mov_b32 %0, m0\n\ts_mov_b32 m0, %6\n\ts_nop 0\n\t"
                     "global_load_lds_dwordx4 %1, %5\n\t"
                     "global_load_lds_dwordx4 %2, %5 offset:1024\n\t"
                     "global_load_lds_dwordx4 %3, %5 offset:2048\n\t"
                     "global_load_lds_dwordx4 %4, %5 offset:3072\n\t"
                     "s_mov_b32 m0, %0"
                     : "=&s"(keep) : "v"(voff[0]), "v"(voff[1]), "v"(voff[2]), "v"(voff[3]), "s"(g), "s"(l) : "memory");
    };
    if (wid != 0) { stageB(0, 0); stageB(1, STG); }

    unsigned* tmo = (unsigned*)(myws + WS_TMO);
    if (wid == 0) {
        const unsigned* f0 = (const unsigned*)(myws + WS_FLAG) + peer * GRID + rg * 64 + lane;
        const unsigned long long t0 = __builtin_amdgcn_s_memrealtime();
        unsigned bad = 0;
        for (unsigned it = 1;; ++it) {
            const unsigned v0 = __hip_atomic_load(f0, __ATOMIC_RELAXED, __HIP_MEMORY_SCOPE_SYSTEM);
            if (__all(v0 >= call)) break;
            if ((it & 63u) == 0u) {
                if (__builtin_amdgcn_readfirstlane(__hip_atomic_load(tmo, __ATOMIC_RELAXED, __HIP_MEMORY_SCOPE_AGENT)) == call
                    || __builtin_amdgcn_s_memrealtime() - t0 > DEADLINE_TICKS) {
                    __hip_atomic_store(tmo, call, __ATOMIC_RELAXED, __HIP_MEMORY_SCOPE_AGENT);
                    bad = 1u;
                    break;
                }
            }
            __builtin_amdgcn_s_sleep(1);
        }
        STAMP(2);
        *(volatile LAS unsigned*)(lds + D * STG) = bad;
        asm volatile("s_waitcnt vmcnt(0) lgkmcnt(0)" ::: "memory");
    }
    __builtin_amdgcn_s_barrier();
    asm volatile("" ::: "memory");
    const bool bad = __builtin_amdgcn_readfirstlane(*(volatile LAS unsigned*)(lds + D * STG)) != 0u;
    STAMP(3);

    const int wm = wid >> 2, wn = wid & 3;
    float* outp = a.out + (size_t)(peer * MS + rg * 128 + wm * 64 + (lane & 15)) * NS + cg * 256 + wn * 64 + (lane >> 4) * 4;
    if (bad) {
        wait_vm<0>();
        const float q = __builtin_nanf("");
        for (int i = 0; i < 4; ++i)
            for (int j = 0; j < 4; ++j)
                *(f32x4*)(outp + (size_t)(i * 16) * NS + j * 16) = (f32x4){q, q, q, q};
        return;
    }
    stageA(0, 0); stageA(1, STG);
    if (wid == 0) { stageB(0, 0); stageB(1, STG); }

    const int fr = lane & 15, fq = lane >> 4;
    const unsigned aoff0 = wm * 8192 + fr * 128 + ((fq ^ (fr >> 1)) << 4), aoff1 = aoff0 ^ 64u;
    const int q8 = (lane & 15) >> 1;
    const int fb = q8 | ((fq & 1) << 3);
    unsigned bo[4];
#pragma unroll
    for (int j = 0; j < 4; ++j) bo[j] = IMG + (fq * 16 + q8) * 256 + (((wn * 4 + j) ^ fb) << 4) + (lane & 1) * 8;

    i32x4 acc[4][4];
#pragma unroll
    for (int i = 0; i < 4; ++i)
#pragma unroll
        for (int j = 0; j < 4; ++j) acc[i][j] = (i32x4){0, 0, 0, 0};

    unsigned sb = 0, sn = 2 * STG;
    for (int s = 0; s < KSTEPS; ++s) {
        if (s == 0) { if (wid == 0) wait_vm<4>(); else wait_vm<2>(); }
        else if (s == KSTEPS - 1) wait_vm<0>();
        else wait_vm<6>();
        __builtin_amdgcn_s_barrier();
        asm volatile("" ::: "memory");
        if (s + 2 < KSTEPS) { stageB(s + 2, sn); stageA(s + 2, sn); }
        i32x4 af[4][2], bf[4][2];
#pragma unroll
        for (int kk = 0; kk < 2; ++kk) {
#pragma unroll
            for (int j = 0; j < 4; ++j) {
                const unsigned ba = bo[j] + sb + kk * 16384;
                const i32x2 lo = __builtin_amdgcn_ds_read_tr8_b64_v2i32((LAS i32x2*)(lds + ba));
                const i32x2 hi = __builtin_amdgcn_ds_read_tr8_b64_v2i32((LAS i32x2*)(lds + ba + 2048));
                bf[j][kk] = (i32x4){lo.x, lo.y, hi.x, hi.y};
            }
#pragma unroll
            for (int i = 0; i < 4; ++i) af[i][kk] = *(const LAS i32x4*)(lds + (kk ? aoff1 : aoff0) + sb + i * 2048);
        }
#pragma unroll
        for (int kk = 0; kk < 2; ++kk)
#pragma unroll
            for (int i = 0; i < 4; ++i)
#pragma unroll
                for (int j = 0; j < 4; ++j)
                    acc[i][j] = __builtin_amdgcn_mfma_i32_16x16x64_i8(bf[j][kk], af[i][kk], acc[i][j], 0, 0, 0);
        sb = (sb == (D - 1) * STG) ? 0u : sb + STG;
        sn = (sn == (D - 1) * STG) ? 0u : sn + STG;
    }
    STAMP(4);
#pragma unroll
    for (int i = 0; i < 4; ++i)
#pragma unroll
        for (int j = 0; j < 4; ++j) {
            f32x4 y;
#pragma unroll
            for (int e = 0; e < 4; ++e) y[e] = fmaxf((float)acc[i][j][e] * scale, 0.0f);
            store_out(outp + (size_t)(i * 16) * NS + j * 16, y);
        }
}

}

extern "C" void kernel_launch(int rank, int nranks, ncclComm_t comm,
                              hipStream_t stream,
                              void* const* d_in, const int* in_sizes, int n_in,
                              void* d_out, int out_size,
                              const int* in_dtypes, int out_dtype,
                              void* const* d_out_all,
                              void* const* d_ws_all, size_t ws_bytes) {
    if (nranks != NR || (unsigned)rank >= (unsigned)NR || n_in != 4 || ws_bytes < WS_NEED ||
        in_sizes[0] != MS * KD || in_sizes[1] != KD * ND || out_size != NR * MS * NS) {
        fprintf(stderr, "kernel_launch: unexpected shapes\n");
        return;
    }
    Args a;
    a.x = (const signed char*)d_in[0]; a.w = (const signed char*)d_in[1];
    a.sx = (const float*)d_in[2]; a.sw = (const float*)d_in[3];
    a.out = (float*)d_out; a.rank = rank; a.pad = 0;
    for (int p = 0; p < NR; ++p) a.ws_all[p] = (char*)d_ws_all[p];
    hipLaunchKernelGGL(gemm_gather_kernel, dim3(GRID), dim3(BLOCK), 0, stream, a);
    HIP_CHECK(hipGetLastError());
}
```

```cpp
#include <hip/hip_runtime.h>
#include <rccl/rccl.h>
#include <cstdio>

#define HIP_CHECK(expr) do { hipError_t _e = (expr); if (_e != hipSuccess) { \
    fprintf(stderr, "HIP error %s at %s:%d\n", hipGetErrorString(_e), __FILE__, __LINE__); \
    return; } } while (0)

namespace {

#define LAS __attribute__((address_space(3)))
#define GAS __attribute__((address_space(1)))
typedef unsigned u32x4 __attribute__((ext_vector_type(4)));
typedef int i32x4 __attribute__((ext_vector_type(4)));
typedef int i32x2 __attribute__((ext_vector_type(2)));
typedef int i32x16 __attribute__((ext_vector_type(16)));

constexpr int NR = 8;
constexpr int KD = 4096, ND = 16384;
constexpr int MS = 512, NS = 2048;
constexpr int GRID = 256, BLOCK = 768;
constexpr int IMG = 16384;
constexpr int STG = 3 * IMG;
constexpr int KSTEPS = KD / 128;
constexpr int D = 3;
constexpr int LDS_SCR = D * STG + 1024;
constexpr int LDS_BYTES = LDS_SCR + 4096;
constexpr int FSTR = 32;

#define P1 100
#define P2 200
#define P3 300
#define S1 20
#define S2 17
#define S3 12
#define DONA 0
#define DONIDX(i, n) (i)
#define DON0 0
#define DON1 0

constexpr size_t WS_CNT = 0;
constexpr size_t WS_TMO = 4096;
constexpr size_t WS_FLAG = 8192;
constexpr size_t WS_CLAIM = 320u << 10;
constexpr size_t WS_X = 1u << 20;
constexpr size_t XBYTES = (size_t)MS * KD;
constexpr size_t WS_NEED = WS_X + 2 * XBYTES;
static_assert(WS_FLAG + (size_t)NR * GRID * FSTR * 4 <= WS_CLAIM && WS_CLAIM + (size_t)GRID * 128 <= WS_X, "layout");
constexpr unsigned long long DEADLINE_TICKS = 50000000ull;

struct Args {
    const signed char* x; char* myws; const float* sx; const float* sw;
    const signed char* w; float* out; int rank; unsigned xmap;
    unsigned long long* stamps;
    char* ws_all[NR];
};

template <int N> __device__ __forceinline__ void wait_vm() { asm volatile("s_waitcnt vmcnt(%0)" :: "n"(N) : "memory"); }
__device__ __forceinline__ void bar() { asm volatile("" ::: "memory"); __builtin_amdgcn_s_barrier(); asm volatile("" ::: "memory"); }
#define STAMP(i) do {} while (0)

__device__ __forceinline__ bool wait_flags(const unsigned* f, unsigned call, unsigned* tmo) {
    const unsigned long long t0 = __builtin_amdgcn_s_memrealtime();
    for (unsigned it = 1;; ++it) {
        const unsigned v = __hip_atomic_load(f, __ATOMIC_RELAXED, __HIP_MEMORY_SCOPE_SYSTEM);
        if (__all(v >= call)) return false;
        if ((it & 63u) == 0u) {
            if (__builtin_amdgcn_readfirstlane(__hip_atomic_load(tmo, __ATOMIC_RELAXED, __HIP_MEMORY_SCOPE_AGENT)) == call
                || __builtin_amdgcn_s_memrealtime() - t0 > DEADLINE_TICKS) {
                __hip_atomic_store(tmo, call, __ATOMIC_RELAXED, __HIP_MEMORY_SCOPE_AGENT);
                return true;
            }
        }
        __builtin_amdgcn_s_sleep(1);
    }
}

__device__ __forceinline__ void warm_row(unsigned& sink, const char* p) {
    asm volatile("s_load_dword %0, %1, 0x0\n\ts_load_dword %0, %1, 0x40\n\ts_load_dword %0, %1, 0x80\n\ts_load_dword %0, %1, 0xc0"
                 : "+s"(sink) : "s"(p) : "memory");
}

struct Frags { i32x4 a[2][2], b[2][2]; };

__global__ __launch_bounds__(BLOCK, 3) void gemm_gather_kernel(Args a) {
    __shared__ __attribute__((aligned(16))) char smem[LDS_BYTES];
    LAS char* lds = (LAS char*)smem;
    const int tid = threadIdx.x, lane = tid & 63, wid = __builtin_amdgcn_readfirstlane(tid >> 6);
    const int b = blockIdx.x, rank = a.rank;
    char* myws = a.myws;
    STAMP(0);
    asm volatile("" :: "s"(a.x), "s"(a.myws), "s"(a.w), "s"(a.rank), "s"(a.xmap), "s"(a.ws_all[0]), "s"(a.ws_all[1]), "s"(a.ws_all[2]),
                 "s"(a.ws_all[3]), "s"(a.ws_all[4]), "s"(a.ws_all[5]), "s"(a.ws_all[6]), "s"(a.ws_all[7]));
    STAMP(10);
    volatile LAS unsigned* ctl = (volatile LAS unsigned*)(lds + D * STG);
    unsigned* tmo = (unsigned*)(myws + WS_TMO);
    const unsigned* cntp = (const unsigned*)(myws + WS_CNT) + b;
    const char* wq = (const char*)a.w + (size_t)rank * NS + (b >> 5) * 256 + (size_t)(((b >> 3) & 3) * 32) * ND;

    if (__builtin_expect(wid < 8, 1)) {
        unsigned call, sink = 0;
        unsigned wlo = 0, whi = 0;
        {
            const int rg_ = b >> 6, ks_ = (b >> 1) & 31, p = (b & 1) * 8192 + tid * 16;
            const int row = p >> 7, chunk = ((p >> 4) & 7) ^ ((row >> 1) & 7);
            const signed char* src = a.x + (size_t)(rg_ * 128 + row) * KD + ks_ * 128 + chunk * 16;
            u32x4 v;
            asm volatile("global_load_dwordx4 %0, %2, off\n\tglobal_load_dword %1, %3, %4 sc1"
                         : "=&v"(v), "=&v"(call) : "v"(src), "v"(0), "s"(cntp) : "memory");
#pragma unroll
        for (int p = 0; p < NR; ++p)
            asm volatile("v_writelane_b32 %0, %2, %4\n\tv_writelane_b32 %1, %3, %4" : "+v"(wlo), "+v"(whi)
                         : "s"((unsigned)(size_t)a.ws_all[p]), "s"((unsigned)((size_t)a.ws_all[p] >> 32)), "n"(p));
            asm volatile("s_waitcnt vmcnt(0)" : "+v"(v), "+v"(call) :: "memory");
            STAMP(11);
            call = __builtin_amdgcn_readfirstlane(call) + 1u;
            char* dst = myws + WS_X + (size_t)(call & 1u) * XBYTES + (size_t)(rg_ * KSTEPS + ks_) * IMG + p;
            asm volatile("global_store_dwordx4 %0, %1, off sc0 sc1\n\ts_nop 1" :: "v"(dst), "v"(v) : "memory");
        }
        char* lanews = (char*)(((size_t)whi << 32) | wlo) + WS_FLAG + (size_t)(rank * GRID + b) * FSTR * 4;
        asm volatile("s_waitcnt vmcnt(0)" : "+v"(lanews) :: "memory");
        STAMP(12);
        bar();
        STAMP(1);
        if (wid == 0 && lane < NR) __hip_atomic_store((GAS unsigned*)lanews, call, __ATOMIC_RELAXED, __HIP_MEMORY_SCOPE_SYSTEM);
        const int wm = wid >> 2, wn = wid & 3;
        const unsigned abase = (wm * 64 + (lane & 31)) * 128 + (((lane >> 5) ^ ((lane & 15) >> 1)) << 4);
        const int q8 = (lane & 15) >> 1;
        const unsigned bbase = IMG + ((lane >> 5) * 16 + q8) * 256 + (((wn * 4 + ((lane >> 4) & 1)) ^ (q8 << 1)) << 4) + (lane & 1) * 8;
        unsigned aoff[4], boff[2];
#pragma unroll
        for (int u = 0; u < 4; ++u) aoff[u] = abase ^ (u << 5);
#pragma unroll
        for (int j = 0; j < 2; ++j) boff[j] = bbase ^ (j << 5);

        i32x16 acc[2][2];
#pragma unroll
        for (int i = 0; i < 2; ++i)
#pragma unroll
            for (int j = 0; j < 2; ++j)
#pragma unroll
                for (int e = 0; e < 16; ++e) acc[i][j][e] = 0;

        auto rd = [&](Frags& f, unsigned sb, int h) {
#pragma unroll
            for (int v = 0; v < 2; ++v) {
                const int u = h * 2 + v;
#pragma unroll
                for (int j = 0; j < 2; ++j) {
                    const unsigned ba = boff[j] + sb + u * 8192;
                    const i32x2 lo = __builtin_amdgcn_ds_read_tr8_b64_v2i32((LAS i32x2*)(lds + ba));
                    const i32x2 hi = __builtin_amdgcn_ds_read_tr8_b64_v2i32((LAS i32x2*)(lds + ba + 2048));
                    f.b[v][j] = (i32x4){lo.x, lo.y, hi.x, hi.y};
                }
#pragma unroll
                for (int i = 0; i < 2; ++i) f.a[v][i] = *(const LAS i32x4*)(lds + aoff[u] + sb + i * 4096);
            }
        };
        auto mm = [&](const Frags& f) {
#pragma unroll
            for (int v = 0; v < 2; ++v)
#pragma unroll
                for (int i = 0; i < 2; ++i)
#pragma unroll
                    for (int j = 0; j < 2; ++j)
                        acc[i][j] = __builtin_amdgcn_mfma_i32_32x32x32_i8(f.a[v][i], f.b[v][j], acc[i][j], 0, 0, 0);
        };
        auto sched = [&]() {
#pragma unroll
            for (int i = 0; i < 6; ++i) { __builtin_amdgcn_sched_group_barrier(0x100, 2, 0); __builtin_amdgcn_sched_group_barrier(0x8, 1, 0); }
            __builtin_amdgcn_sched_group_barrier(0x8, 2, 0);
        };
        Frags f0, f1;
        unsigned sb = 0;
        float scale;
        {
            float s0, s1;
            asm volatile("s_load_dword %0, %3, 0x0\n\ts_load_dword %1, %4, 0x0\n\ts_waitcnt lgkmcnt(0)"
                         : "=&s"(s0), "=&s"(s1), "+s"(sink) : "s"(a.sx), "s"(a.sw) : "memory");
            scale = s0 * s1;
        }
        bar();
        if (tid == 0) __hip_atomic_store((unsigned*)(myws + WS_CNT) + b, call, __ATOMIC_RELAXED, __HIP_MEMORY_SCOPE_AGENT);
        const unsigned slot = __builtin_amdgcn_readfirstlane(ctl[1]);
        const int peer = (a.xmap >> (4 * (slot >> 5))) & 7, rg = slot & 3, cg = (slot >> 2) & 7;
        bar();
        STAMP(3);
        rd(f0, sb, 0);
        for (int s = 0; s < KSTEPS - 2; ++s) {
            __builtin_amdgcn_sched_barrier(0);
            rd(f1, sb, 1); mm(f0); sched();
            __builtin_amdgcn_sched_barrier(0);
            asm volatile("s_waitcnt lgkmcnt(0)" ::: "memory");
            bar();
            sb = (sb == (D - 1) * STG) ? 0u : sb + STG;
            rd(f0, sb, 0); mm(f1); sched();
        }
        __builtin_amdgcn_sched_barrier(0);
        rd(f1, sb, 1); mm(f0); sched();
        __builtin_amdgcn_sched_barrier(0);
        mm(f1);
        __builtin_amdgcn_sched_barrier(0);
        bar();
        sb = (sb == (D - 1) * STG) ? 0u : sb + STG;
        rd(f0, sb, 0); rd(f1, sb, 1);
        const bool bad = __builtin_amdgcn_readfirstlane(ctl[0]) != 0u;
        mm(f0); mm(f1);
        STAMP(4);
        const char* ob = (const char*)(a.out + (size_t)(peer * MS + rg * 128 + wm * 64) * NS + cg * 256 + wn * 64);
        const unsigned ov = (lane >> 5) * 4 * NS * 4 + (lane & 31) * 4;
        const unsigned sbits = __builtin_amdgcn_readfirstlane(__float_as_uint(scale));
        if (!bad && (sbits & 0x7f800000u) != 0x7f800000u) {
#pragma unroll
            for (int i = 0; i < 2; ++i)
#pragma unroll
                for (int r = 0; r < 16; ++r) {
                    const char* rb = ob + (size_t)(i * 32 + (r & 3) + 8 * (r >> 2)) * NS * 4;
                    const float y0 = fmaxf((float)acc[i][0][r] * scale, 0.0f), y1 = fmaxf((float)acc[i][1][r] * scale, 0.0f);
                    asm volatile("global_store_dword %0, %1, %3 sc1\n\tglobal_store_dword %0, %2, %3 offset:128 sc1"
                                 :: "v"(ov), "v"(y0), "v"(y1), "s"(rb) : "memory");
                }
        } else {
            if (bad) scale = __builtin_nanf("");
#pragma unroll
            for (int i = 0; i < 2; ++i)
#pragma unroll
                for (int r = 0; r < 16; ++r) {
                    const char* rb = ob + (size_t)(i * 32 + (r & 3) + 8 * (r >> 2)) * NS * 4;
                    float y0 = (float)acc[i][0][r] * scale, y1 = (float)acc[i][1][r] * scale;
                    y0 = (y0 < 0.0f) ? 0.0f : y0; y1 = (y1 < 0.0f) ? 0.0f : y1;
                    asm volatile("global_store_dword %0, %1, %3 sc1\n\tglobal_store_dword %0, %2, %3 offset:128 sc1"
                                 :: "v"(ov), "v"(y0), "v"(y1), "s"(rb) : "memory");
                }
        }
    } else {
        const int lw = wid - 8;
        unsigned* claim = (unsigned*)(myws + WS_CLAIM);
        const unsigned slot0 = ((unsigned)__builtin_amdgcn_s_getreg(20 | (3 << 11)) & 7u) * 32u + (unsigned)(b >> 3);
        unsigned call, old = 0;
        asm volatile("global_load_dword %0, %1, %2 sc1" : "=v"(call) : "v"(0), "s"(cntp) : "memory");
        if (lw == 0 && lane == 0)
            asm volatile("global_atomic_add %0, %1, %2, %3 sc0" : "=v"(old) : "v"(0), "v"(1), "s"(claim + slot0 * 32) : "memory");
        const unsigned ldsA = (unsigned)(size_t)lds + lw * 4096, ldsB = (unsigned)(size_t)lds + IMG + lw * 8192;
        const unsigned voffA = lane * 16;
        unsigned voff[2];
#pragma unroll
        for (int i = 0; i < 2; ++i) { const int k = i * 4 + (lane >> 4);
            voff[i] = k * ND + (((lane & 15) ^ (k << 1)) << 4); }
        const char* gW = (const char*)a.w + (size_t)rank * NS + (size_t)(lw * 32) * ND;
        unsigned sink = 0;
        STAMP(8);
        bar();
        asm volatile("s_waitcnt vmcnt(0) lgkmcnt(0)" : "+v"(call), "+v"(old), "+s"(sink) :: "memory");
        call = __builtin_amdgcn_readfirstlane(call) + 1u;
        STAMP(9);
        unsigned slot = slot0;
        if (lw == 0) {
            unsigned lost = 0;
            if (lane == 0) {
#pragma nounroll
                for (unsigned tries = 0; old != call - 1u; ++tries) {
                    __hip_atomic_fetch_sub(claim + slot * 32, 1u, __ATOMIC_RELAXED, __HIP_MEMORY_SCOPE_AGENT);
                    if (tries >= GRID) { lost = 1u; break; }
                    slot = (slot + 1u) & (GRID - 1);
                    old = __hip_atomic_fetch_add(claim + slot * 32, 1u, __ATOMIC_RELAXED, __HIP_MEMORY_SCOPE_AGENT);
                }
            }
            slot = __builtin_amdgcn_readfirstlane(slot);
            lost = __builtin_amdgcn_readfirstlane(lost);
            const int pr = (a.xmap >> (4 * (slot >> 5))) & 7;
            if (wait_flags((const unsigned*)(myws + WS_FLAG) + (pr * GRID + (slot & 3) * 64 + lane) * FSTR, call, tmo)) lost = 1u;
            ctl[0] = lost; ctl[1] = slot;
            asm volatile("s_waitcnt vmcnt(0) lgkmcnt(0)" ::: "memory");
        }
        STAMP(2);
        bar();
        const bool bad = __builtin_amdgcn_readfirstlane(ctl[0]) != 0u;
        slot = __builtin_amdgcn_readfirstlane(ctl[1]);
        const int peer = (a.xmap >> (4 * (slot >> 5))) & 7, rg = slot & 3, cg = (slot >> 2) & 7;
        const char* gB = gW + cg * 256;
        const char* gX = a.ws_all[bad ? rank : peer] + WS_X + (size_t)(call & 1u) * XBYTES;
        const char* gA = gX + (size_t)rg * KSTEPS * IMG + lw * 4096;
        auto ld1 = [&](const char* g, unsigned l, unsigned v) {
            unsigned keep;
            asm volatile("s_mov_b32 %0, m0\n\ts_mov_b32 m0, %3\n\ts_nop 0\n\tglobal_load_lds_dwordx4 %1, %2\n\ts_mov_b32 m0, %0"
                         : "=&s"(keep) : "v"(v), "s"(g), "s"(l) : "memory");
        };
        auto ldB = [&](const char* g, unsigned slotoff, int i) {
            ld1(g + (size_t)(i * 8) * ND, ldsB + slotoff + i * 2048, voff[0]);
            ld1(g + (size_t)(i * 8) * ND, ldsB + slotoff + i * 2048 + 1024, voff[1]);
        };
        auto stage = [&](int s, unsigned so) {
            const char* g = gB + (size_t)s * 128 * ND;
            const char* ga = gA + (size_t)s * IMG;
#pragma unroll
            for (int i = 0; i < 4; ++i) { ldB(g, so, i); ld1(ga + i * 1024, ldsA + so + i * 1024, voffA); }
        };
#pragma unroll
        for (int i = 0; i < 4; ++i) ld1(gA + i * 1024, ldsA + i * 1024, voffA);
#pragma unroll
        for (int i = 0; i < 4; ++i) ldB(gB, 0, i);
        stage(1, STG); stage(2, 2 * STG);
        wait_vm<24>();
        bar();
        unsigned sn = 0;
        bool don_ok = false;
        const int pstep = (peer == rank || rg == 0) ? -1 : (rg == 1 ? S1 : rg == 2 ? S2 : S3);
        const unsigned plen = rg == 1 ? P1 : rg == 2 ? P2 : P3;
        for (int s = 1; s < KSTEPS - 2; ++s) {
            wait_vm<12>();
            bar();
            if ((P1 | P2 | P3) != 0 && s == pstep) {
                const unsigned long long t0 = __builtin_amdgcn_s_memrealtime();
                while (__builtin_amdgcn_s_memrealtime() - t0 < plen) __builtin_amdgcn_s_sleep(1);
            }
            if ((DON0 | DON1) != 0 && s == 8 && rg < 2 && lw < 2) {
                const unsigned v = __hip_atomic_load((const unsigned*)(myws + WS_FLAG) + (peer * GRID + (3 - rg) * 64 + lane) * FSTR, __ATOMIC_RELAXED, __HIP_MEMORY_SCOPE_SYSTEM);
                don_ok = __all(v >= call);
            }
            stage(s + 2, sn);
            sn = (sn == (D - 1) * STG) ? 0u : sn + STG;
        }
        const int drg = 3 - rg, dn = (peer == rank || lw >= 2 || bad || !don_ok) ? 0 : rg == 0 ? DON0 : rg == 1 ? DON1 : 0;
        const char* gd = gX + (size_t)(drg * KSTEPS + KSTEPS - 1) * IMG + (cg * 2 + lw) * 1024;
        wait_vm<12>();
        bar();
        if (dn > 0 && DONA > 0) {
#pragma unroll
            for (int i = 0; i < DONA; ++i) ld1(gd - (size_t)DONIDX(i, dn) * IMG, (unsigned)(size_t)lds + LDS_SCR + lw * 1024, voffA);
            wait_vm<DONA>();
        } else wait_vm<0>();
        bar();
        for (int i = DONA; i < dn; ++i) ld1(gd - (size_t)DONIDX(i, dn) * IMG, (unsigned)(size_t)lds + LDS_SCR + lw * 1024, voffA);
        wait_vm<0>();
    }
}

}

extern "C" void kernel_launch(int rank, int nranks, ncclComm_t comm,
                              hipStream_t stream,
                              void* const* d_in, const int* in_sizes, int n_in,
                              void* d_out, int out_size,
                              const int* in_dtypes, int out_dtype,
                              void* const* d_out_all,
                              void* const* d_ws_all, size_t ws_bytes) {
    if (nranks != NR || (unsigned)rank >= (unsigned)NR || n_in != 4 || ws_bytes < WS_NEED ||
        in_sizes[0] != MS * KD || in_sizes[1] != KD * ND || out_size != NR * MS * NS) {
        fprintf(stderr, "kernel_launch: unexpected shapes\n");
        return;
    }
    Args a;
    a.x = (const signed char*)d_in[0]; a.w = (const signed char*)d_in[1];
    a.sx = (const float*)d_in[2]; a.sw = (const float*)d_in[3];
    a.out = (float*)d_out; a.rank = rank; a.myws = (char*)d_ws_all[rank]; a.stamps = nullptr;
    static const unsigned kXmap[NR] = {0x74513206u, 0x26305417u, 0x17546023u, 0x47150632u,
                                       0x36027145u, 0x60231754u, 0x51472360u, 0x02364571u};
    a.xmap = kXmap[rank];
    for (int p = 0; p < NR; ++p) a.ws_all[p] = (char*)d_ws_all[p];
    hipLaunchKernelGGL(gemm_gather_kernel, dim3(GRID), dim3(BLOCK), 0, stream, a);
    HIP_CHECK(hipGetLastError());
}
```

```cpp
#include <hip/hip_runtime.h>
#include <rccl/rccl.h>
#include <cstdio>

#define HIP_CHECK(expr) do { hipError_t _e = (expr); if (_e != hipSuccess) { \
    fprintf(stderr, "HIP error %s at %s:%d\n", hipGetErrorString(_e), __FILE__, __LINE__); \
    return; } } while (0)

namespace {

#define LAS __attribute__((address_space(3)))
#define GAS __attribute__((address_space(1)))
typedef unsigned u32x4 __attribute__((ext_vector_type(4)));
typedef int i32x4 __attribute__((ext_vector_type(4)));
typedef int i32x2 __attribute__((ext_vector_type(2)));
typedef int i32x16 __attribute__((ext_vector_type(16)));

constexpr int NR = 8;
constexpr int KD = 4096, ND = 16384;
constexpr int MS = 512, NS = 2048;
constexpr int GRID = 256, BLOCK = 768;
constexpr int IMG = 16384;
constexpr int STG = 3 * IMG;
constexpr int KSTEPS = KD / 128;
constexpr int D = 3;
constexpr int LDS_SCR = D * STG + 1024;
constexpr int LDS_ALT = LDS_SCR + 2048;
constexpr int LDS_BYTES = LDS_ALT + 8192;
constexpr int FSTR = 32;

#define P1 0
#define P2 0
#define P3 0
#define S1 20
#define S2 17
#define S3 12
#define LM0 0x3ffffffeu
#define LM1 0u
#define LM2 0u
#define LM3 0u
#define NFIRST 6
#define INITV 3
#define PPOLL 0
#define TAILV 3
#define PACE 0
#define M16 1
#define Q1 9
#define Q2 13
#define Q3 16
#define LDIST 3

constexpr size_t WS_CNT = 0;
constexpr size_t WS_TMO = 4096;
constexpr size_t WS_FLAG = 8192;
constexpr size_t WS_CLAIM = 320u << 10;
constexpr size_t WS_X = 1u << 20;
constexpr size_t XBYTES = (size_t)MS * KD;
constexpr size_t WS_NEED = WS_X + 2 * XBYTES;
static_assert(WS_FLAG + (size_t)NR * GRID * FSTR * 4 <= WS_CLAIM && WS_CLAIM + (size_t)GRID * 128 <= WS_X, "layout");
constexpr unsigned long long DEADLINE_TICKS = 50000000ull;

struct Args {
    const signed char* x; char* myws; const float* sx; const float* sw;
    const signed char* w; float* out; int rank; unsigned xmap;
    unsigned long long* stamps;
    char* ws_all[NR];
};

template <int N> __device__ __forceinline__ void wait_vm() { asm volatile("s_waitcnt vmcnt(%0)" :: "n"(N) : "memory"); }
__device__ __forceinline__ void bar() { asm volatile("" ::: "memory"); __builtin_amdgcn_s_barrier(); asm volatile("" ::: "memory"); }
#define STAMP(i) do {} while (0)
#define TSTAMP(i) do {} while (0)

__device__ __forceinline__ bool wait_flags(const unsigned* f, unsigned call, unsigned* tmo) {
    const unsigned long long t0 = __builtin_amdgcn_s_memrealtime();
    for (unsigned it = 1;; ++it) {
        const unsigned v = __hip_atomic_load(f, __ATOMIC_RELAXED, __HIP_MEMORY_SCOPE_SYSTEM);
        if (__all(v >= call)) return false;
        if ((it & 63u) == 0u) {
            if (__builtin_amdgcn_readfirstlane(__hip_atomic_load(tmo, __ATOMIC_RELAXED, __HIP_MEMORY_SCOPE_AGENT)) == call
                || __builtin_amdgcn_s_memrealtime() - t0 > DEADLINE_TICKS) {
                __hip_atomic_store(tmo, call, __ATOMIC_RELAXED, __HIP_MEMORY_SCOPE_AGENT);
                return true;
            }
        }
        __builtin_amdgcn_s_sleep(1);
    }
}

struct Frags { i32x4 a[2][2], b[2][2]; };

__global__ __launch_bounds__(BLOCK, 3) void gemm_gather_kernel(Args a) {
    __shared__ __attribute__((aligned(16))) char smem[LDS_BYTES];
    LAS char* lds = (LAS char*)smem;
    const int tid = threadIdx.x, lane = tid & 63, wid = __builtin_amdgcn_readfirstlane(tid >> 6);
    const int b = blockIdx.x, rank = a.rank;
    char* myws = a.myws;
    STAMP(0);
    volatile LAS unsigned* ctl = (volatile LAS unsigned*)(lds + D * STG);
    asm volatile("" :: "s"(a.x), "s"(a.myws), "s"(a.w), "s"(a.rank), "s"(a.xmap), "s"(a.ws_all[0]), "s"(a.ws_all[1]), "s"(a.ws_all[2]),
                 "s"(a.ws_all[3]), "s"(a.ws_all[4]), "s"(a.ws_all[5]), "s"(a.ws_all[6]), "s"(a.ws_all[7]));
    STAMP(10);
    unsigned* tmo = (unsigned*)(myws + WS_TMO);
    const unsigned* cntp = (const unsigned*)(myws + WS_CNT) + b;

    if (__builtin_expect(wid < 8, 1)) {
        unsigned call;
        unsigned wlo = 0, whi = 0;
        {
            const int rg_ = b >> 6, ks_ = (b >> 1) & 31, p = (b & 1) * 8192 + tid * 16;
            const int row = p >> 7, chunk = ((p >> 4) & 7) ^ ((row >> 1) & 7);
            const signed char* src = a.x + (size_t)(rg_ * 128 + row) * KD + ks_ * 128 + chunk * 16;
            u32x4 v;
            asm volatile("global_load_dwordx4 %0, %2, off\n\tglobal_load_dword %1, %3, %4 sc1"
                         : "=&v"(v), "=&v"(call) : "v"(src), "v"(0), "s"(cntp) : "memory");
#pragma unroll
        for (int p = 0; p < NR; ++p)
            asm volatile("v_writelane_b32 %0, %2, %4\n\tv_writelane_b32 %1, %3, %4" : "+v"(wlo), "+v"(whi)
                         : "s"((unsigned)(size_t)a.ws_all[p]), "s"((unsigned)((size_t)a.ws_all[p] >> 32)), "n"(p));
            bar();
            asm volatile("s_waitcnt vmcnt(0)" : "+v"(v), "+v"(call) :: "memory");
            STAMP(11);
            call = __builtin_amdgcn_readfirstlane(call) + 1u;
            char* dst = myws + WS_X + (size_t)(call & 1u) * XBYTES + (size_t)(rg_ * KSTEPS + ks_) * IMG + p;
            asm volatile("global_store_dwordx4 %0, %1, off sc0 sc1\n\ts_nop 1" :: "v"(dst), "v"(v) : "memory");
        }
        char* lanews = (char*)(((size_t)whi << 32) | wlo) + WS_FLAG + (size_t)(rank * GRID + b) * FSTR * 4;
        asm volatile("s_waitcnt vmcnt(0)" : "+v"(lanews) :: "memory");
        STAMP(12);
        {
            unsigned arrived = 0;
            if (lane == 0) arrived = __hip_atomic_fetch_add((LAS unsigned*)(lds + D * STG + 8), 1u, __ATOMIC_RELAXED, __HIP_MEMORY_SCOPE_WORKGROUP);
            if (__builtin_amdgcn_readfirstlane(arrived) == 7u) {
                if (lane < NR) __hip_atomic_store((GAS unsigned*)lanews, call, __ATOMIC_RELAXED, __HIP_MEMORY_SCOPE_SYSTEM);
            }
        }
        STAMP(1);
        const int wm = wid >> 2, wn = wid & 3;
        const int q8 = (lane & 15) >> 1;
        unsigned aoff[2], boff[4];
        aoff[0] = (wm * 64 + (lane & 15)) * 128 + (((lane >> 4) ^ q8) << 4); aoff[1] = aoff[0] ^ 64u;
#pragma unroll
        for (int j = 0; j < 4; ++j) boff[j] = IMG + ((lane >> 4) * 16 + q8) * 256 + (((wn * 4 + j) ^ (q8 | (((lane >> 4) & 1) << 3))) << 4) + (lane & 1) * 8;
        i32x4 acc[4][4];
#pragma unroll
        for (int i = 0; i < 4; ++i)
#pragma unroll
            for (int j = 0; j < 4; ++j) acc[i][j] = (i32x4){0, 0, 0, 0};
        auto rd2 = [&](Frags& f, unsigned sa, unsigned sb, int h) {
#pragma unroll
            for (int t = 0; t < 4; ++t) {
                const unsigned ba = boff[t] + sb + h * 16384;
                const i32x2 lo = __builtin_amdgcn_ds_read_tr8_b64_v2i32((LAS i32x2*)(lds + ba));
                const i32x2 hi = __builtin_amdgcn_ds_read_tr8_b64_v2i32((LAS i32x2*)(lds + ba + 2048));
                f.b[t >> 1][t & 1] = (i32x4){lo.x, lo.y, hi.x, hi.y};
                f.a[t >> 1][t & 1] = *(const LAS i32x4*)(lds + aoff[h] + sa + t * 2048);
            }
        };
        auto rd = [&](Frags& f, unsigned sb, int h) { rd2(f, sb, sb, h); };
        auto mm = [&](const Frags& f) {
#pragma unroll
            for (int i = 0; i < 4; ++i)
#pragma unroll
                for (int j = 0; j < 4; ++j)
                    acc[i][j] = __builtin_amdgcn_mfma_i32_16x16x64_i8(f.a[i >> 1][i & 1], f.b[j >> 1][j & 1], acc[i][j], 0, 0, 0);
        };
        auto sched = [&]() {
#pragma unroll
            for (int i = 0; i < 12; ++i) { __builtin_amdgcn_sched_group_barrier(0x100, 1, 0); __builtin_amdgcn_sched_group_barrier(0x8, 1, 0); }
            __builtin_amdgcn_sched_group_barrier(0x8, 4, 0);
        };
        Frags f0, f1;
        unsigned long long clk = 0; (void)clk;
        unsigned sb = 0;
        float scale;
        {
            float s0, s1;
            asm volatile("s_load_dword %0, %2, 0x0\n\ts_load_dword %1, %3, 0x0\n\ts_waitcnt lgkmcnt(0)"
                         : "=&s"(s0), "=&s"(s1) : "s"(a.sx), "s"(a.sw) : "memory");
            scale = s0 * s1;
        }
        bar();
        if (tid == 0) __hip_atomic_store((unsigned*)(myws + WS_CNT) + b, call, __ATOMIC_RELAXED, __HIP_MEMORY_SCOPE_AGENT);
        const unsigned slot = __builtin_amdgcn_readfirstlane(ctl[1]);
        const int peer = (a.xmap >> (4 * (slot >> 5))) & 7, rg = slot & 3, cg = (slot >> 2) & 7;
        const char* ob = (const char*)(a.out + (size_t)(peer * MS + rg * 128 + wm * 64) * NS + cg * 256 + wn * 64);
        asm volatile("" :: "s"(ob));
        if (NFIRST < 64 && wid == 1) {
            __builtin_amdgcn_s_sleep(12);
            if (wait_flags((const unsigned*)(myws + WS_FLAG) + (peer * GRID + rg * 64 + lane) * FSTR, call, tmo)) ctl[0] = 1u;
            asm volatile("s_waitcnt vmcnt(0) lgkmcnt(0)" ::: "memory");
        }
        bar();
        STAMP(3);
        rd(f0, sb, 0);
        for (int s = 0; s < KSTEPS - 4; ++s) {
            __builtin_amdgcn_sched_barrier(0);
            rd(f1, sb, 1); mm(f0); sched();
            __builtin_amdgcn_sched_barrier(0);
            asm volatile("s_waitcnt lgkmcnt(0)" ::: "memory");
            bar();
            sb = (sb == (D - 1) * STG) ? 0u : sb + STG;
            rd(f0, sb, 0); mm(f1); sched();
        }
        unsigned S27 = (27 % D) * STG, S28 = (28 % D) * STG, S29 = (29 % D) * STG, S30 = (30 % D) * STG, S31 = (31 % D) * STG;
        asm volatile("" : "+s"(S27), "+s"(S28), "+s"(S29), "+s"(S30), "+s"(S31));
        if (__builtin_amdgcn_readfirstlane(ctl[0]) != 0u) scale = __builtin_nanf("");
        unsigned ov = 0;
        auto st2 = [&](int i2, int r) {
            const int i = i2 * 2 + (r >> 3), jp = (r >> 2) & 1, e = r & 3;
            const auto sw = __builtin_amdgcn_permlane16_swap((unsigned)acc[i][2 * jp][e], (unsigned)acc[i][2 * jp + 1][e], false, false);
            const float y0 = __builtin_elementwise_maximum((float)(int)sw[0] * scale, 0.0f);
            const float y1 = __builtin_elementwise_maximum((float)(int)sw[1] * scale, 0.0f);
            const char* rb = ob + (size_t)(i * 16 + e) * NS * 4 + jp * 128;
            asm volatile("global_store_dword %0, %1, %3 sc1\n\tglobal_store_dword %0, %2, %4 sc1"
                         :: "v"(ov), "v"(y0), "v"(y1), "s"(rb), "s"(rb + 4 * NS * 4) : "memory");
        };
        auto mm1 = [&](int i2, int q) {
            const Frags& f = q < 4 ? f0 : f1;
#pragma unroll
            for (int t = 0; t < 2; ++t)
                acc[i2 * 2 + t][q & 3] = __builtin_amdgcn_mfma_i32_16x16x64_i8(f.a[i2][t], f.b[(q >> 1) & 1][q & 1], acc[i2 * 2 + t][q & 3], 0, 0, 0);
        };
        auto rdB = [&](Frags& f, unsigned so, int h) {
#pragma unroll
            for (int t = 0; t < 4; ++t) {
                const unsigned ba = boff[t] + so + h * 16384;
                const i32x2 lo = __builtin_amdgcn_ds_read_tr8_b64_v2i32((LAS i32x2*)(lds + ba));
                const i32x2 hi = __builtin_amdgcn_ds_read_tr8_b64_v2i32((LAS i32x2*)(lds + ba + 2048));
                f.b[t >> 1][t & 1] = (i32x4){lo.x, lo.y, hi.x, hi.y};
            }
        };
        auto rdA = [&](Frags& f, unsigned so, int h, int i2) {
#pragma unroll
            for (int t = 0; t < 2; ++t) f.a[i2][t] = *(const LAS i32x4*)(lds + aoff[h] + so + (i2 * 2 + t) * 2048);
        };
        auto mmh = [&](const Frags& f, int i2) {
#pragma unroll
            for (int t = 0; t < 2; ++t)
#pragma unroll
                for (int j = 0; j < 4; ++j)
                    acc[i2 * 2 + t][j] = __builtin_amdgcn_mfma_i32_16x16x64_i8(f.a[i2][t], f.b[j >> 1][j & 1], acc[i2 * 2 + t][j], 0, 0, 0);
        };
#define SB0 __builtin_amdgcn_sched_barrier(0)
        asm volatile("" : "+v"(scale));
        SB0; rd(f1, S28, 1); mm(f0); sched(); SB0;
        asm volatile("s_waitcnt lgkmcnt(0)" ::: "memory");
        bar();
        STAMP(5);
        rdA(f0, S29, 0, 0); rdB(f0, S29, 0); SB0; mm(f1); SB0;
        rdA(f1, S29, 1, 0); rdB(f1, S29, 1); SB0; mmh(f0, 0); SB0;
        bar();
        rdA(f0, S30, 0, 0); rdB(f0, S30, 0); SB0; mmh(f1, 0); SB0;
        rdA(f1, S30, 1, 0); rdB(f1, S30, 1); SB0; mmh(f0, 0); SB0;
        bar();
        STAMP(10);
        rdA(f0, S31, 0, 0); rdB(f0, S31, 0); SB0; mmh(f1, 0); SB0;
        rdA(f1, S31, 1, 0); rdB(f1, S31, 1); SB0; mmh(f0, 0); SB0;
        rdB(f0, S29, 0); SB0; mmh(f1, 0); SB0;
        rdB(f1, S29, 1); SB0;
        {
            unsigned l2;
            asm volatile("v_mbcnt_lo_u32_b32 %0, -1, 0\n\tv_mbcnt_hi_u32_b32 %0, -1, %0" : "=v"(l2));
            ov = (l2 >> 5) * 8 * NS * 4 + (l2 & 31) * 4;
        }
        STAMP(4);
#pragma unroll
        for (int r = 0; r < Q1; ++r) { st2(0, r); SB0; }
        bar();
        rdA(f0, S29, 0, 1); rdA(f1, S29, 1, 1); SB0; mmh(f0, 1); SB0; rdB(f0, S30, 0); SB0; mmh(f1, 1); SB0; rdB(f1, S30, 1); SB0;
#pragma unroll
        for (int r = Q1; r < Q2; ++r) { st2(0, r); SB0; }
        bar();
        rdA(f0, S30, 0, 1); rdA(f1, S30, 1, 1); SB0; mmh(f0, 1); SB0; rdB(f0, S31, 0); SB0; mmh(f1, 1); SB0; rdB(f1, S31, 1); SB0;
#pragma unroll
        for (int r = Q2; r < Q3; ++r) { st2(0, r); SB0; }
        bar();
        STAMP(11);
        rdA(f0, S31, 0, 1); rdA(f1, S31, 1, 1); SB0; mmh(f0, 1); mmh(f1, 1); SB0;
        STAMP(12);
#pragma unroll
        for (int r = Q3; r < 32; ++r) { st2(r >> 4, r & 15); SB0; }
#undef SB0
    } else {
        const int lw = wid - 8;
        unsigned* claim = (unsigned*)(myws + WS_CLAIM);
        const unsigned slot0 = ((unsigned)__builtin_amdgcn_s_getreg(20 | (3 << 11)) & 7u) * 32u + (unsigned)(b >> 3);
        unsigned call, old = 0;
        asm volatile("global_load_dword %0, %1, %2 sc1" : "=v"(call) : "v"(0), "s"(cntp) : "memory");
        if (lw == 0 && lane == 0)
            asm volatile("global_atomic_add %0, %1, %2, %3 sc0" : "=v"(old) : "v"(0), "v"(1), "s"(claim + slot0 * 32) : "memory");
        const unsigned ldsA = (unsigned)(size_t)lds + lw * 4096, ldsB = (unsigned)(size_t)lds + IMG + lw * 8192;
        const unsigned voffA = lane * 16;
        unsigned voff[2];
#pragma unroll
        for (int i = 0; i < 2; ++i) { const int k = i * 4 + (lane >> 4);
            voff[i] = k * ND + (((lane & 15) ^ (M16 ? k : (k << 1))) << 4); }
        const unsigned voffx[2] = {M16 ? voff[0] ^ 128u : voff[0], M16 ? voff[1] ^ 128u : voff[1]};
        const char* gW = (const char*)a.w + (size_t)rank * NS + (size_t)(lw * 32) * ND;
        auto ld1 = [&](const char* g, unsigned l, unsigned v) {
            unsigned keep;
            asm volatile("s_mov_b32 %0, m0\n\ts_mov_b32 m0, %3\n\ts_nop 0\n\tglobal_load_lds_dwordx4 %1, %2\n\ts_mov_b32 m0, %0"
                         : "=&s"(keep) : "v"(v), "s"(g), "s"(l) : "memory");
        };
        if (lw == 0 && lane < 12) ctl[lane] = 0u;
        STAMP(8);
        asm volatile("s_waitcnt lgkmcnt(0)" ::: "memory");
        bar();
        asm volatile("s_waitcnt vmcnt(0) lgkmcnt(0)" : "+v"(call), "+v"(old) :: "memory");
        call = __builtin_amdgcn_readfirstlane(call) + 1u;
        unsigned slot = slot0;
        unsigned pv0 = 0, pv1 = 0, pv2 = 0, pv3 = 0;
        if (lw == 0) {
            unsigned lost = 0;
            if (lane == 0) {
#pragma nounroll
                for (unsigned tries = 0; old != call - 1u; ++tries) {
                    __hip_atomic_fetch_sub(claim + slot * 32, 1u, __ATOMIC_RELAXED, __HIP_MEMORY_SCOPE_AGENT);
                    if (tries >= GRID) { lost = 1u; break; }
                    slot = (slot + 1u) & (GRID - 1);
                    old = __hip_atomic_fetch_add(claim + slot * 32, 1u, __ATOMIC_RELAXED, __HIP_MEMORY_SCOPE_AGENT);
                }
            }
            slot = __builtin_amdgcn_readfirstlane(slot);
            lost = __builtin_amdgcn_readfirstlane(lost);
            const int pr = (a.xmap >> (4 * (slot >> 5))) & 7;
            const char* g0 = a.ws_all[pr] + WS_X + (size_t)(call & 1u) * XBYTES + (size_t)(slot & 3) * KSTEPS * IMG;
            asm volatile("" :: "s"(g0));
            ctl[1] = slot;
            bool seen = false;
            if (PPOLL) {
                const unsigned* fp = (const unsigned*)(myws + WS_FLAG) + (pr * GRID + (slot & 3) * 64 + (lane < NFIRST ? lane : 0)) * FSTR;
                unsigned v0, v1, v2, v3;
                asm volatile("global_load_dword %0, %4, off sc0 sc1\n\ts_sleep %5\n\tglobal_load_dword %1, %4, off sc0 sc1\n\ts_sleep %5\n\t"
                             "global_load_dword %2, %4, off sc0 sc1\n\ts_sleep %5\n\tglobal_load_dword %3, %4, off sc0 sc1"
                             : "=&v"(v0), "=&v"(v1), "=&v"(v2), "=&v"(v3) : "v"(fp), "n"(PPOLL) : "memory");
                for (int it = 0; it < 64; ++it) {
#define POLL_STEP(v) asm volatile("s_waitcnt vmcnt(3)" : "+v"(v) :: "memory"); if (__all(v >= call)) { seen = true; break; } \
                    asm volatile("global_load_dword %0, %1, off sc0 sc1" : "=v"(v) : "v"(fp) : "memory");
                    POLL_STEP(v0) POLL_STEP(v1) POLL_STEP(v2) POLL_STEP(v3)
#undef POLL_STEP
                }
                if (!seen) asm volatile("s_waitcnt vmcnt(0)" : "+v"(v0), "+v"(v1), "+v"(v2), "+v"(v3) :: "memory");
                pv0 = v0; pv1 = v1; pv2 = v2; pv3 = v3;
            }
            if (!seen && wait_flags((const unsigned*)(myws + WS_FLAG) + (pr * GRID + (slot & 3) * 64 + (lane < NFIRST ? lane : 0)) * FSTR, call, tmo)) lost = 1u;
            STAMP(2);
            if (lost) ctl[0] = 1u;
            else {
#pragma unroll
                for (int i = 0; i < 16; ++i)
                    if ((i & 3) < (INITV == 0 || INITV == 1 ? 4 : INITV == 3 ? 2 : INITV == 4 ? 3 : 1)) ld1(g0 + i * 1024, ldsA + i * 1024, voffA);
            }
            asm volatile("s_waitcnt lgkmcnt(0)" ::: "memory");
        }
        bar();
        bool bad = __builtin_amdgcn_readfirstlane(ctl[0]) != 0u;
        slot = __builtin_amdgcn_readfirstlane(ctl[1]);
        const int peer = (a.xmap >> (4 * (slot >> 5))) & 7, rg = slot & 3, cg = (slot >> 2) & 7;
        const char* gB = gW + cg * 256;
        const char* gX = a.ws_all[bad ? rank : peer] + WS_X + (size_t)(call & 1u) * XBYTES;
        const char* gA = gX + (size_t)rg * KSTEPS * IMG + lw * 4096;
        auto ldB = [&](const char* g, unsigned slotoff, int i) {
            ld1(g + (size_t)(i * 8) * ND, ldsB + slotoff + i * 2048, (i & 2) ? voffx[0] : voff[0]);
            ld1(g + (size_t)(i * 8) * ND, ldsB + slotoff + i * 2048 + 1024, (i & 2) ? voffx[1] : voff[1]);
        };
        auto stage = [&](int s, unsigned so) {
            const char* g = gB + (size_t)s * 128 * ND;
            const char* ga = gA + (size_t)s * IMG;
#pragma unroll
            for (int i = 0; i < 4; ++i) { ldB(g, so, i); ld1(ga + i * 1024, ldsA + so + i * 1024, voffA); }
        };
        auto ldA = [&](int n) { ld1(gA + (size_t)(n >> 2) * IMG + (n & 3) * 1024, ldsA + (n >> 2) * STG + (n & 3) * 1024, voffA); };
        auto ldBn = [&](int n) { ldB(gB + (size_t)(n >> 2) * 128 * ND, (n >> 2) * STG, n & 3); };
        if (INITV == 0) {
#pragma unroll
            for (int i = 0; i < 4; ++i) ldB(gB, 0, i);
            stage(1, STG); stage(2, 2 * STG);
            wait_vm<24>();
        } else if (INITV == 1) {
            ldBn(0); ldBn(1); ldA(4); ldBn(2); ldA(5); ldBn(3);
            ldBn(4); ldA(6); ldBn(5); ldA(7); ldBn(6); ldBn(7);
            ldA(8); ldBn(8); ldA(9); ldBn(9); ldBn(10); ldA(10); ldBn(11); ldA(11);
            wait_vm<22>();
        } else if (INITV == 3) {
            ldBn(0); ldA(2); ldBn(1); ldA(3); ldBn(2); ldA(4); ldBn(3);
            ldA(5); ldBn(4); ldA(6); ldBn(5); ldBn(6); ldA(7); ldBn(7);
            ldA(8); ldBn(8); ldA(9); ldBn(9); ldBn(10); ldA(10); ldBn(11); ldA(11);
            wait_vm<23>();
        } else if (INITV == 4) {
            ldBn(0); ldA(3); ldBn(1); ldBn(2); ldA(4); ldBn(3);
            ldA(5); ldBn(4); ldA(6); ldBn(5); ldBn(6); ldA(7); ldBn(7);
            ldA(8); ldBn(8); ldA(9); ldBn(9); ldBn(10); ldA(10); ldBn(11); ldA(11);
            wait_vm<23>();
        } else {
            ldBn(0); ldA(1); ldBn(1); ldA(2); ldBn(2); ldA(3); ldBn(3);
            ldA(4); ldBn(4); ldA(5); ldBn(5); ldA(6); ldBn(6); ldA(7); ldBn(7);
            ldA(8); ldBn(8); ldA(9); ldBn(9); ldBn(10); ldA(10); ldBn(11); ldA(11);
            wait_vm<24>();
        }
        asm volatile("" :: "v"(pv0), "v"(pv1), "v"(pv2), "v"(pv3));
        bar();
        if (__builtin_amdgcn_readfirstlane(ctl[0]) != 0u) { gX = myws + WS_X; gA = gX + lw * 4096; }
        unsigned sn = 0;
        const int pstep = (peer == rank || rg == 0) ? -1 : (rg == 1 ? S1 : rg == 2 ? S2 : S3);
        const unsigned plen = rg == 1 ? P1 : rg == 2 ? P2 : P3;
        const unsigned lmask = (peer == rank || lw >= 2) ? 0u : (rg == 0 ? LM0 : rg == 1 ? LM1 : rg == 2 ? LM2 : LM3);
        const char* gL = gX + (size_t)(rg * KSTEPS + LDIST) * IMG + (cg * 2 + lw) * 1024;
        bool led = false;
        for (int s = 1; s < KSTEPS - 5; ++s) {
            if (led) wait_vm<13>(); else wait_vm<12>();
            bar();
            if ((P1 | P2 | P3) != 0 && s == pstep) {
                const unsigned long long t0 = __builtin_amdgcn_s_memrealtime();
                while (__builtin_amdgcn_s_memrealtime() - t0 < plen) __builtin_amdgcn_s_sleep(1);
            }
            led = (LM0 | LM1 | LM2 | LM3) != 0u && ((lmask >> s) & 1u) && s + LDIST < KSTEPS;
            if (led) ld1(gL + (size_t)s * IMG, (unsigned)(size_t)lds + LDS_SCR + lw * 1024, voffA);
            stage(s + 2, sn);
            sn = (sn == (D - 1) * STG) ? 0u : sn + STG;
        }
        const unsigned hoff = (lw >> 1) * 8192 + (lw & 1) * 2048;
        const char* gH = gA - lw * 4096 + hoff;
        const unsigned ldsH = (unsigned)(size_t)lds + hoff;
        auto ldH = [&](int s, int i) {
            ld1(gH + (size_t)s * IMG + i * 4096, ldsH + (s % D) * STG + i * 4096, voffA);
            ld1(gH + (size_t)s * IMG + i * 4096 + 1024, ldsH + (s % D) * STG + i * 4096 + 1024, voffA);
        };
        auto ldBs = [&](int s) {
#pragma unroll
            for (int i = 0; i < 4; ++i) ldB(gB + (size_t)s * 128 * ND, (s % D) * STG, i);
        };
        auto lead = [&](int s) {
            led = (LM0 | LM1 | LM2 | LM3) != 0u && ((lmask >> s) & 1u) && s + LDIST < KSTEPS;
            if (led) ld1(gL + (size_t)s * IMG, (unsigned)(size_t)lds + LDS_SCR + lw * 1024, voffA);
        };
        if (led) wait_vm<13>(); else wait_vm<12>();
        bar();
        lead(27); ldH(29, 0); ldBs(29);
        if (led) wait_vm<11>(); else wait_vm<10>();
        bar();
        lead(28); ldH(30, 0); ldBs(30);
        if (led) wait_vm<11>(); else wait_vm<10>();
        bar();
        ldH(31, 0);
        if (TAILV == 2) ldH(29, 1);
        ldBs(31);
        if (TAILV == 3) {
            wait_vm<10>();
            bar();
            ldH(29, 1); ldH(30, 1); ldH(31, 1);
            wait_vm<6>();
        } else {
            if (TAILV == 0) ldH(29, 1);
            ldH(30, 1); ldH(31, 1);
            wait_vm<16>();
            bar();
            if (TAILV == 0) wait_vm<6>(); else wait_vm<4>();
        }
        bar();
        wait_vm<4>();
        bar();
        wait_vm<2>();
        bar();
        wait_vm<0>();
        STAMP(9);
        bar();
    }
}

}

extern "C" void kernel_launch(int rank, int nranks, ncclComm_t comm,
                              hipStream_t stream,
                              void* const* d_in, const int* in_sizes, int n_in,
                              void* d_out, int out_size,
                              const int* in_dtypes, int out_dtype,
                              void* const* d_out_all,
                              void* const* d_ws_all, size_t ws_bytes) {
    if (nranks != NR || (unsigned)rank >= (unsigned)NR || n_in != 4 || ws_bytes < WS_NEED ||
        in_sizes[0] != MS * KD || in_sizes[1] != KD * ND || out_size != NR * MS * NS) {
        fprintf(stderr, "kernel_launch: unexpected shapes\n");
        return;
    }
    Args a;
    a.x = (const signed char*)d_in[0]; a.w = (const signed char*)d_in[1];
    a.sx = (const float*)d_in[2]; a.sw = (const float*)d_in[3];
    a.out = (float*)d_out; a.rank = rank; a.myws = (char*)d_ws_all[rank]; a.stamps = nullptr;
    static const unsigned kXmap[NR] = {0x74513206u, 0x26305417u, 0x17546023u, 0x47150632u,
                                       0x36027145u, 0x60231754u, 0x51472360u, 0x02364571u};
    a.xmap = kXmap[rank];
    for (int p = 0; p < NR; ++p) a.ws_all[p] = (char*)d_ws_all[p];
    hipLaunchKernelGGL(gemm_gather_kernel, dim3(GRID), dim3(BLOCK), 0, stream, a);
    HIP_CHECK(hipGetLastError());
}
```

```cpp
#include <hip/hip_runtime.h>
#include <rccl/rccl.h>
#include <cstdio>

#define HIP_CHECK(expr) do { hipError_t _e = (expr); if (_e != hipSuccess) { \
    fprintf(stderr, "HIP error %s at %s:%d\n", hipGetErrorString(_e), __FILE__, __LINE__); \
    return; } } while (0)

namespace {

#define LAS __attribute__((address_space(3)))
#define GAS __attribute__((address_space(1)))
typedef unsigned u32x4 __attribute__((ext_vector_type(4)));
typedef int i32x4 __attribute__((ext_vector_type(4)));
typedef int i32x2 __attribute__((ext_vector_type(2)));

constexpr int NR = 8;
constexpr int KD = 4096, ND = 16384;
constexpr int MS = 512, NS = 2048;
constexpr int GRID = 256, BLOCK = 768;
constexpr int IMG = 16384;
constexpr int STG = 3 * IMG;
constexpr int KSTEPS = KD / 128;
constexpr int D = 3;
constexpr int LDS_CTL = D * STG;
constexpr int LDS_SCR = LDS_CTL + 1024;
constexpr int LDS_BYTES = LDS_SCR + 2048;
constexpr int NFIRST = 6;

constexpr size_t WS_CNT = 0;
constexpr size_t WS_TMO = 4096;
constexpr size_t WS_FLAG = 8192;
constexpr size_t WS_CLAIM = 32768;
constexpr size_t WS_X = 1u << 20;
constexpr size_t XBYTES = (size_t)MS * KD;
constexpr size_t WS_NEED = WS_X + 2 * XBYTES;
static_assert(WS_FLAG + (size_t)NR * 8 * 128 <= WS_CLAIM && WS_CLAIM + (size_t)GRID * 128 <= WS_X, "layout");
constexpr unsigned long long DEADLINE_TICKS = 50000000ull;

struct Args {
    const signed char* x; char* myws; const float* sx; const float* sw;
    const signed char* w; float* out; int rank; unsigned xmap;
    unsigned long long* stamps;
    char* ws_all[NR];
};

template <int N> __device__ __forceinline__ void wait_vm() { asm volatile("s_waitcnt vmcnt(%0)" :: "n"(N) : "memory"); }
__device__ __forceinline__ void bar() { asm volatile("" ::: "memory"); __builtin_amdgcn_s_barrier(); asm volatile("" ::: "memory"); }
#define SB0 __builtin_amdgcn_sched_barrier(0)
#define INL __attribute__((always_inline))
#define STAMP(i) do {} while (0)

__device__ __forceinline__ bool wait_count(const char* f, int lane, unsigned want, unsigned call, unsigned* tmo) {
    const unsigned long long t0 = __builtin_amdgcn_s_memrealtime();
    for (unsigned it = 1;; ++it) {
        unsigned v = 0;
        if (lane == 0) asm volatile("global_atomic_add %0, %1, %1, %2 sc0 sc1\n\ts_waitcnt vmcnt(0)" : "=&v"(v) : "v"(0), "s"(f) : "memory");
        if (__builtin_amdgcn_readfirstlane(v) >= want) return false;
        if ((it & 63u) == 0u) {
            if (__builtin_amdgcn_readfirstlane(__hip_atomic_load(tmo, __ATOMIC_RELAXED, __HIP_MEMORY_SCOPE_AGENT)) == call
                || __builtin_amdgcn_s_memrealtime() - t0 > DEADLINE_TICKS) {
                __hip_atomic_store(tmo, call, __ATOMIC_RELAXED, __HIP_MEMORY_SCOPE_AGENT);
                return true;
            }
        }
        if (it > 64u) __builtin_amdgcn_s_sleep(4);
    }
}

struct Frags { i32x4 a[4], b[4]; };

__global__ __launch_bounds__(BLOCK, 3) void gemm_gather_kernel(Args a) {
    __shared__ __attribute__((aligned(16))) char smem[LDS_BYTES];
    LAS char* lds = (LAS char*)smem;
    const int tid = threadIdx.x, lane = tid & 63, wid = __builtin_amdgcn_readfirstlane(tid >> 6);
    const int b = blockIdx.x, rank = a.rank;
    char* myws = a.myws;
    STAMP(0);
    volatile LAS unsigned* ctl = (volatile LAS unsigned*)(lds + LDS_CTL);
    asm volatile("" :: "s"(a.x), "s"(a.myws), "s"(a.w), "s"(a.rank), "s"(a.xmap), "s"(a.ws_all[0]), "s"(a.ws_all[1]), "s"(a.ws_all[2]),
                 "s"(a.ws_all[3]), "s"(a.ws_all[4]), "s"(a.ws_all[5]), "s"(a.ws_all[6]), "s"(a.ws_all[7]));
    SB0;
    unsigned* tmo = (unsigned*)(myws + WS_TMO);
    const unsigned* cntp = (const unsigned*)(myws + WS_CNT) + b;

    if (__builtin_expect(wid < 8, 1)) {
        unsigned call;
        {
            const int prg = (b >> 1) & 3, pks = b >> 3, p = (b & 1) * 8192 + tid * 16;
            const int row = p >> 7, chunk = ((p >> 4) & 7) ^ ((row >> 1) & 7);
            const signed char* src = a.x + (size_t)(prg * 128 + row) * KD + pks * 128 + chunk * 16;
            if (pks >= NFIRST / 2) __builtin_amdgcn_s_sleep(8);
            u32x4 v;
            asm volatile("global_load_dwordx4 %0, %2, off\n\tglobal_load_dword %1, %3, %4 sc1"
                         : "=&v"(v), "=&v"(call) : "v"(src), "v"(0), "s"(cntp) : "memory");
            unsigned wlo = 0, whi = 0;
#pragma unroll
            for (int q = 0; q < NR; ++q)
                asm volatile("v_writelane_b32 %0, %2, %4\n\tv_writelane_b32 %1, %3, %4" : "+v"(wlo), "+v"(whi)
                             : "s"((unsigned)(size_t)a.ws_all[q]), "s"((unsigned)((size_t)a.ws_all[q] >> 32)), "n"(q));
            bar();
            asm volatile("s_waitcnt vmcnt(0)" : "+v"(v), "+v"(call) :: "memory");
            call = __builtin_amdgcn_readfirstlane(call) + 1u;
            char* dst = myws + WS_X + (size_t)(call & 1u) * XBYTES + (size_t)(prg * KSTEPS + pks) * IMG + p;
            asm volatile("global_store_dwordx4 %0, %1, off sc0 sc1\n\ts_nop 1" :: "v"(dst), "v"(v) : "memory");
            char* flag = (char*)(((size_t)whi << 32) | wlo) + WS_FLAG + (size_t)((rank * 4 + prg) * 2 + (pks >= NFIRST / 2)) * 128;
            asm volatile("s_waitcnt vmcnt(0)" : "+v"(flag) :: "memory");
            unsigned arrived = 0;
            if (lane == 0) arrived = __hip_atomic_fetch_add((LAS unsigned*)(lds + LDS_CTL + 8), 1u, __ATOMIC_RELAXED, __HIP_MEMORY_SCOPE_WORKGROUP);
            if (__builtin_amdgcn_readfirstlane(arrived) == 7u && lane < NR)
                asm volatile("global_atomic_add %0, %1, off sc1" :: "v"(flag), "v"(1) : "memory");
        }
        STAMP(1);
        const int wm = wid >> 2, wn = wid & 3;
        const int q8 = (lane & 15) >> 1;
        unsigned aoff[2], boff[4];
        aoff[0] = (wm * 64 + (lane & 15)) * 128 + (((lane >> 4) ^ q8) << 4); aoff[1] = aoff[0] ^ 64u;
#pragma unroll
        for (int j = 0; j < 4; ++j)
            boff[j] = IMG + ((lane >> 4) * 16 + q8) * 256 + (((wn * 4 + j) ^ (q8 | (((lane >> 4) & 1) << 3))) << 4) + (lane & 1) * 8;
        i32x4 acc[4][4];
#pragma unroll
        for (int i = 0; i < 4; ++i)
#pragma unroll
            for (int j = 0; j < 4; ++j) acc[i][j] = (i32x4){0, 0, 0, 0};

        auto rdB1 = [&](Frags& f, unsigned so, int h, int j) INL {
            const unsigned ba = boff[j] + so + h * 16384;
            const i32x2 lo = __builtin_amdgcn_ds_read_tr8_b64_v2i32((LAS i32x2*)(lds + ba));
            const i32x2 hi = __builtin_amdgcn_ds_read_tr8_b64_v2i32((LAS i32x2*)(lds + ba + 2048));
            f.b[j] = (i32x4){lo.x, lo.y, hi.x, hi.y};
        };
        auto rdA1 = [&](Frags& f, unsigned so, int h, int i) INL { f.a[i] = *(const LAS i32x4*)(lds + aoff[h] + so + i * 2048); };
        auto rd = [&](Frags& f, unsigned so, int h) INL {
#pragma unroll
            for (int t = 0; t < 4; ++t) { rdB1(f, so, h, t); rdA1(f, so, h, t); }
        };
        auto rdB = [&](Frags& f, unsigned so, int h) INL {
#pragma unroll
            for (int t = 0; t < 4; ++t) rdB1(f, so, h, t);
        };
        auto rdA = [&](Frags& f, unsigned so, int h, int half) INL { rdA1(f, so, h, 2 * half); rdA1(f, so, h, 2 * half + 1); };
        auto mmr = [&](const Frags& f, int i) INL {
#pragma unroll
            for (int j = 0; j < 4; ++j) acc[i][j] = __builtin_amdgcn_mfma_i32_16x16x64_i8(f.a[i], f.b[j], acc[i][j], 0, 0, 0);
        };
        auto mmh = [&](const Frags& f, int half) INL { mmr(f, 2 * half); mmr(f, 2 * half + 1); };
        auto mm = [&](const Frags& f) INL { mmh(f, 0); mmh(f, 1); };
        auto sched = [&]() INL {
#pragma unroll
            for (int i = 0; i < 12; ++i) { __builtin_amdgcn_sched_group_barrier(0x100, 1, 0); __builtin_amdgcn_sched_group_barrier(0x8, 1, 0); }
            __builtin_amdgcn_sched_group_barrier(0x8, 4, 0);
        };
        float scale;
        {
            float s0, s1;
            asm volatile("s_load_dword %0, %2, 0x0\n\ts_load_dword %1, %3, 0x0\n\ts_waitcnt lgkmcnt(0)"
                         : "=&s"(s0), "=&s"(s1) : "s"(a.sx), "s"(a.sw) : "memory");
            scale = s0 * s1;
        }
        bar();
        if (tid == 0) __hip_atomic_store((unsigned*)(myws + WS_CNT) + b, call, __ATOMIC_RELAXED, __HIP_MEMORY_SCOPE_AGENT);
        const unsigned slot = __builtin_amdgcn_readfirstlane(ctl[1]);
        const int peer = (a.xmap >> (4 * (slot >> 5))) & 7, rg = slot & 3, cg = (slot >> 2) & 7;
        const char* ob = (const char*)(a.out + (size_t)(peer * MS + rg * 128 + wm * 64) * NS + cg * 256 + wn * 64);
        asm volatile("" :: "s"(ob));
        if (wid == 1) {
            __builtin_amdgcn_s_sleep(20);
            if (wait_count(myws + WS_FLAG + ((peer * 4 + rg) * 2 + 1) * 128, lane, call * (64 - NFIRST), call, tmo)) ctl[0] = 1u;
            asm volatile("s_waitcnt vmcnt(0) lgkmcnt(0)" ::: "memory");
        }
        Frags f0, f1;
        unsigned sb = 0;
        bar();
        STAMP(3);
        rd(f0, sb, 0);
        for (int s = 0; s < KSTEPS - 4; ++s) {
            SB0; rd(f1, sb, 1); mm(f0); sched(); SB0;
            asm volatile("s_waitcnt lgkmcnt(0)" ::: "memory");
            bar();
            sb = (sb == (D - 1) * STG) ? 0u : sb + STG;
            rd(f0, sb, 0); mm(f1); sched();
        }
        unsigned S28 = (28 % D) * STG, S29 = (29 % D) * STG, S30 = (30 % D) * STG, S31 = (31 % D) * STG;
        asm volatile("" : "+s"(S28), "+s"(S29), "+s"(S30), "+s"(S31));
        if (__builtin_amdgcn_readfirstlane(ctl[0]) != 0u) scale = __builtin_nanf("");
        asm volatile("" : "+v"(scale));
        SB0; rd(f1, S28, 1); mm(f0); sched(); SB0;
        asm volatile("s_waitcnt lgkmcnt(0)" ::: "memory");
        bar();
        STAMP(5);
        rdA(f0, S29, 0, 0); rdB(f0, S29, 0); SB0; mm(f1); SB0;
        rdA(f1, S29, 1, 0); rdB(f1, S29, 1); SB0; mmh(f0, 0); SB0;
        bar();
        rdA(f0, S30, 0, 0); rdB(f0, S30, 0); SB0; mmh(f1, 0); SB0;
        rdA(f1, S30, 1, 0); rdB(f1, S30, 1); SB0; mmh(f0, 0); SB0;
        bar();
        rdA(f0, S31, 0, 0); rdB(f0, S31, 0); SB0; mmh(f1, 0); SB0;
        rdA(f1, S31, 1, 0); rdB(f1, S31, 1); SB0; mmh(f0, 0); SB0;
        rdB(f0, S29, 0); SB0; mmh(f1, 0); SB0;
        rdB(f1, S29, 1); SB0;
        unsigned ov;
        asm volatile("v_mbcnt_lo_u32_b32 %0, -1, 0\n\tv_mbcnt_hi_u32_b32 %0, -1, %0" : "=v"(ov));
        ov = (ov >> 5) * 8 * NS * 4 + (ov & 31) * 4;
        auto st = [&](int n) INL {
            const int i = n >> 3, jp = (n >> 2) & 1, e = n & 3;
            const auto sw = __builtin_amdgcn_permlane16_swap((unsigned)acc[i][2 * jp][e], (unsigned)acc[i][2 * jp + 1][e], false, false);
            const float y0 = __builtin_elementwise_maximum((float)(int)sw[0] * scale, 0.0f);
            const float y1 = __builtin_elementwise_maximum((float)(int)sw[1] * scale, 0.0f);
            const char* rb = ob + (size_t)(i * 16 + e) * NS * 4 + jp * 128;
            asm volatile("global_store_dword %0, %1, %3 sc1\n\tglobal_store_dword %0, %2, %4 sc1"
                         :: "v"(ov), "v"(y0), "v"(y1), "s"(rb), "s"(rb + 4 * NS * 4) : "memory");
        };
        STAMP(4);
#pragma unroll
        for (int n = 0; n < 9; ++n) { st(n); SB0; }
        bar();
        rdA(f0, S29, 0, 1); rdA(f1, S29, 1, 1); SB0; mmh(f0, 1); SB0; rdB(f0, S30, 0); SB0; mmh(f1, 1); SB0; rdB(f1, S30, 1); SB0;
#pragma unroll
        for (int n = 9; n < 13; ++n) { st(n); SB0; }
        bar();
        rdA(f0, S30, 0, 1); rdA(f1, S30, 1, 1); SB0; mmh(f0, 1); SB0; rdB(f0, S31, 0); SB0; mmh(f1, 1); SB0; rdB(f1, S31, 1); SB0;
#pragma unroll
        for (int n = 13; n < 16; ++n) { st(n); SB0; }
        bar();
        rdA(f0, S31, 0, 1); rdA(f1, S31, 1, 1); SB0; mmh(f0, 1); mmh(f1, 1); SB0;
        STAMP(7);
#pragma unroll
        for (int n = 16; n < 32; ++n) { st(n); SB0; }
    } else {
        const int lw = wid - 8;
        unsigned* claim = (unsigned*)(myws + WS_CLAIM);
        unsigned slot = ((unsigned)__builtin_amdgcn_s_getreg(20 | (3 << 11)) & 7u) * 32u + (unsigned)(b >> 3);
        unsigned call, old = 0;
        asm volatile("global_load_dword %0, %1, %2 sc1" : "=v"(call) : "v"(0), "s"(cntp) : "memory");
        if (lw == 0 && lane == 0)
            asm volatile("global_atomic_add %0, %1, %2, %3 sc0" : "=v"(old) : "v"(0), "v"(1), "s"(claim + slot * 32) : "memory");
        const unsigned lds0 = (unsigned)(size_t)lds, ldsA = lds0 + lw * 4096, ldsB = lds0 + IMG + lw * 8192;
        const unsigned voffA = lane * 16;
        unsigned voff[4];
#pragma unroll
        for (int i = 0; i < 4; ++i) { const int k = (i & 1) * 4 + (lane >> 4);
            voff[i] = k * ND + (((lane & 15) ^ (k | ((i >> 1) << 3))) << 4); }
        const char* gW = (const char*)a.w + (size_t)rank * NS + (size_t)(lw * 32) * ND;
        auto ld1 = [&](const char* g, unsigned l, unsigned v) INL {
            unsigned keep;
            asm volatile("s_mov_b32 %0, m0\n\ts_mov_b32 m0, %3\n\ts_nop 0\n\tglobal_load_lds_dwordx4 %1, %2\n\ts_mov_b32 m0, %0"
                         : "=&s"(keep) : "v"(v), "s"(g), "s"(l) : "memory");
        };
        if (lw == 0 && lane < 4) ctl[lane] = 0u;
        asm volatile("s_waitcnt lgkmcnt(0)" ::: "memory");
        bar();
        asm volatile("s_waitcnt vmcnt(0)" : "+v"(call), "+v"(old) :: "memory");
        call = __builtin_amdgcn_readfirstlane(call) + 1u;
        if (lw == 0) {
            unsigned lost = 0;
            if (lane == 0) {
#pragma nounroll
                for (unsigned tries = 0; old != call - 1u; ++tries) {
                    __hip_atomic_fetch_sub(claim + slot * 32, 1u, __ATOMIC_RELAXED, __HIP_MEMORY_SCOPE_AGENT);
                    if (tries >= GRID) { lost = 1u; break; }
                    slot = (slot + 1u) & (GRID - 1);
                    old = __hip_atomic_fetch_add(claim + slot * 32, 1u, __ATOMIC_RELAXED, __HIP_MEMORY_SCOPE_AGENT);
                }
            }
            slot = __builtin_amdgcn_readfirstlane(slot);
            lost = __builtin_amdgcn_readfirstlane(lost);
            const int pr = (a.xmap >> (4 * (slot >> 5))) & 7;
            const char* g0 = a.ws_all[pr] + WS_X + (size_t)(call & 1u) * XBYTES + (size_t)(slot & 3) * KSTEPS * IMG;
            asm volatile("" :: "s"(g0));
            ctl[1] = slot;
            if (wait_count(myws + WS_FLAG + (pr * 4 + (slot & 3)) * 2 * 128, lane, call * NFIRST, call, tmo)) lost = 1u;
            STAMP(2);
            if (lost) ctl[0] = 1u;
            else {
#pragma unroll
                for (int i = 0; i < 8; ++i) ld1(g0 + (i >> 1) * 4096 + (i & 1) * 1024, lds0 + (i >> 1) * 4096 + (i & 1) * 1024, voffA);
            }
            asm volatile("s_waitcnt lgkmcnt(0)" ::: "memory");
        }
        bar();
        const bool bad = __builtin_amdgcn_readfirstlane(ctl[0]) != 0u;
        slot = __builtin_amdgcn_readfirstlane(ctl[1]);
        const int peer = (a.xmap >> (4 * (slot >> 5))) & 7, rg = slot & 3, cg = (slot >> 2) & 7;
        const char* gB = gW + cg * 256;
        const char* gR = a.ws_all[bad ? rank : peer] + WS_X + (size_t)(call & 1u) * XBYTES + (size_t)rg * KSTEPS * IMG;
        auto ldA = [&](int s, unsigned so, int i) INL { ld1(gR + (size_t)s * IMG + lw * 4096 + i * 1024, ldsA + so + i * 1024, voffA); };
        auto ldB = [&](int s, unsigned so, int i) INL {
            const char* g = gB + (size_t)(s * 128 + i * 8) * ND;
            ld1(g, ldsB + so + i * 2048, voff[i & 2]);
            ld1(g, ldsB + so + i * 2048 + 1024, voff[(i & 2) + 1]);
        };
        auto A = [&](int n) INL { ldA(n >> 2, (n >> 2) * STG, n & 3); };
        auto B = [&](int n) INL { ldB(n >> 2, (n >> 2) * STG, n & 3); };
        B(0); A(2); B(1); A(3); B(2); A(4); B(3);
        A(5); B(4); A(6); B(5); B(6); A(7); B(7);
        A(8); B(8); A(9); B(9); B(10); A(10); B(11); A(11);
        wait_vm<23>();
        bar();
        if (__builtin_amdgcn_readfirstlane(ctl[0]) != 0u) gR = myws + WS_X;
        const bool lead = peer != rank && rg == 0 && lw < 2;
        auto early = [&](int s) INL { if (lead) ld1(gR + (size_t)s * IMG + (cg * 2 + lw) * 1024, lds0 + LDS_SCR + lw * 1024, voffA); };
        wait_vm<12>();
        unsigned sn = 0;
        for (int s = 1; s < KSTEPS - 5; ++s) {
            bar();
            early(s + 3);
#pragma unroll
            for (int i = 0; i < 4; ++i) { ldB(s + 2, sn, i); ldA(s + 2, sn, i); }
            sn = (sn == (D - 1) * STG) ? 0u : sn + STG;
            if (lead) wait_vm<13>(); else wait_vm<12>();
        }
        const unsigned hoff = (lw >> 1) * 8192 + (lw & 1) * 2048;
        auto ldH = [&](int s, int half) INL {
            ld1(gR + (size_t)s * IMG + hoff + half * 4096, lds0 + (s % D) * STG + hoff + half * 4096, voffA);
            ld1(gR + (size_t)s * IMG + hoff + half * 4096 + 1024, lds0 + (s % D) * STG + hoff + half * 4096 + 1024, voffA);
        };
        auto ldU = [&](int s) INL {
            ldH(s, 0);
#pragma unroll
            for (int i = 0; i < 4; ++i) ldB(s, (s % D) * STG, i);
        };
        bar();
        early(30); ldU(29);
        if (lead) wait_vm<11>(); else wait_vm<10>();
        bar();
        early(31); ldU(30);
        if (lead) wait_vm<11>(); else wait_vm<10>();
        bar();
        ldU(31);
        wait_vm<10>();
        bar();
        ldH(29, 1); ldH(30, 1); ldH(31, 1);
        wait_vm<6>();
        bar();
        wait_vm<4>();
        bar();
        wait_vm<2>();
        bar();
        wait_vm<0>();
        bar();
    }
}

}

extern "C" void kernel_launch(int rank, int nranks, ncclComm_t comm,
                              hipStream_t stream,
                              void* const* d_in, const int* in_sizes, int n_in,
                              void* d_out, int out_size,
                              const int* in_dtypes, int out_dtype,
                              void* const* d_out_all,
                              void* const* d_ws_all, size_t ws_bytes) {
    if (nranks != NR || (unsigned)rank >= (unsigned)NR || n_in != 4 || ws_bytes < WS_NEED ||
        in_sizes[0] != MS * KD || in_sizes[1] != KD * ND || out_size != NR * MS * NS) {
        fprintf(stderr, "kernel_launch: unexpected shapes\n");
        return;
    }
    Args a;
    a.x = (const signed char*)d_in[0]; a.w = (const signed char*)d_in[1];
    a.sx = (const float*)d_in[2]; a.sw = (const float*)d_in[3];
    a.out = (float*)d_out; a.rank = rank; a.myws = (char*)d_ws_all[rank]; a.stamps = nullptr;
    static const unsigned kXmap[NR] = {0x74513206u, 0x26305417u, 0x17546023u, 0x47150632u,
                                       0x36027145u, 0x60231754u, 0x51472360u, 0x02364571u};
    a.xmap = kXmap[rank];
    for (int p = 0; p < NR; ++p) a.ws_all[p] = (char*)d_ws_all[p];
    hipLaunchKernelGGL(gemm_gather_kernel, dim3(GRID), dim3(BLOCK), 0, stream, a);
    HIP_CHECK(hipGetLastError());
}
```

```cpp
#include <hip/hip_runtime.h>
#include <rccl/rccl.h>
#include <cstdio>

#define HIP_CHECK(expr) do { hipError_t _e = (expr); if (_e != hipSuccess) { \
    fprintf(stderr, "HIP error %s at %s:%d\n", hipGetErrorString(_e), __FILE__, __LINE__); \
    return; } } while (0)

namespace {

#define LAS __attribute__((address_space(3)))
#define GAS __attribute__((address_space(1)))
typedef unsigned u32x4 __attribute__((ext_vector_type(4)));
typedef int i32x4 __attribute__((ext_vector_type(4)));
typedef int i32x2 __attribute__((ext_vector_type(2)));

constexpr int NR = 8;
constexpr int KD = 4096, ND = 16384;
constexpr int MS = 512, NS = 2048;
constexpr int GRID = 256, BLOCK = 768;
constexpr int IMG = 16384;
constexpr int STG = 3 * IMG;
constexpr int KSTEPS = KD / 128;
constexpr int D = 3;
constexpr int LDS_CTL = D * STG;
constexpr int LDS_SCR = LDS_CTL + 1024;
constexpr int LDS_BYTES = LDS_SCR + 2048;
constexpr int NFIRST = 6;

#define LAY 1
#define TW1 2
#define TWL 19
#define CPF 0
#define JMP 1
#define TW0 3
#define TDEL 0
#define TM1 0x100ffu
#define TM2 0x6ff00u
#define LATE 8
#define TW2 1
#define SCL 1

constexpr size_t WS_CNT = 0;
constexpr size_t WS_TMO = 4096;
constexpr size_t WS_FLAG = 8192;
constexpr size_t WS_CLAIM = 32768;
constexpr size_t WS_PAD = 65536;
constexpr size_t WS_X = LAY ? 131072 : 1u << 20;
constexpr size_t XBYTES = (size_t)MS * KD;
constexpr size_t WS_NEED = WS_X + 2 * XBYTES + KSTEPS * 256;
constexpr size_t X_KS = LAY ? 8 * IMG + 256 : IMG,
                 X_PAR = LAY ? 4 * IMG : XBYTES, X_RG = LAY ? IMG : KSTEPS * IMG;
static_assert(WS_FLAG + (size_t)NR * 8 * 128 <= WS_CLAIM && WS_CLAIM + (size_t)GRID * 128 <= WS_PAD, "layout");
constexpr unsigned long long DEADLINE_TICKS = 50000000ull;

struct Args {
    const signed char* x; char* myws; const float* sx; const float* sw;
    const signed char* w; float* out; int rank; unsigned xmap;
    unsigned long long* stamps;
    char* ws_all[NR];
};

template <int N> __device__ __forceinline__ void wait_vm() { asm volatile("s_waitcnt vmcnt(%0)" :: "n"(N) : "memory"); }
__device__ __forceinline__ void bar() { asm volatile("" ::: "memory"); __builtin_amdgcn_s_barrier(); asm volatile("" ::: "memory"); }
#define SB0 __builtin_amdgcn_sched_barrier(0)
#define INL __attribute__((always_inline))
#define STAMP(i) do {} while (0)

__device__ __forceinline__ bool wait_count(const char* f, int lane, unsigned want, unsigned call, unsigned* tmo) {
    const unsigned long long t0 = __builtin_amdgcn_s_memrealtime();
    for (unsigned it = 1;; ++it) {
        unsigned v = 0;
        if (lane == 0) asm volatile("global_atomic_add %0, %1, %1, %2 sc0 sc1\n\ts_waitcnt vmcnt(0)" : "=&v"(v) : "v"(0), "s"(f) : "memory");
        if (__builtin_amdgcn_readfirstlane(v) >= want) return false;
        if ((it & 63u) == 0u) {
            if (__builtin_amdgcn_readfirstlane(__hip_atomic_load(tmo, __ATOMIC_RELAXED, __HIP_MEMORY_SCOPE_AGENT)) == call
                || __builtin_amdgcn_s_memrealtime() - t0 > DEADLINE_TICKS) {
                __hip_atomic_store(tmo, call, __ATOMIC_RELAXED, __HIP_MEMORY_SCOPE_AGENT);
                return true;
            }
        }
        if (it > 64u) __builtin_amdgcn_s_sleep(4);
    }
}

struct Frags { i32x4 a[4], b[4]; };

__global__ __launch_bounds__(BLOCK, 3) void gemm_gather_kernel(Args a) {
    __shared__ __attribute__((aligned(16))) char smem[LDS_BYTES];
    LAS char* lds = (LAS char*)smem;
    const int tid = threadIdx.x, lane = tid & 63, wid = __builtin_amdgcn_readfirstlane(tid >> 6);
    const int b = blockIdx.x;
    STAMP(0);
    volatile LAS unsigned* ctl = (volatile LAS unsigned*)(lds + LDS_CTL);
    unsigned cpf = 0;
    if (CPF) {
        unsigned long long pc;
        unsigned off = ((b >> 3) * 4 + (lane & 3)) * 128;
        asm volatile("s_getpc_b64 %2\n.Lgg_pc:\n\tv_min_u32 %1, .Lgg_end-.Lgg_pc, %1\n\t"
                     "s_cmp_lg_u32 %3, 10\n\ts_cbranch_scc1 1f\n\tglobal_load_dword %0, %1, %2\n1:"
                     : "+v"(cpf), "+v"(off), "=&s"(pc) : "s"(wid) : "scc");
    }
#define KARGS asm volatile("" :: "s"(a.x), "s"(a.myws), "s"(a.w), "s"(a.rank), "s"(a.xmap), "s"(a.ws_all[0]), "s"(a.ws_all[1]), "s"(a.ws_all[2]), \
                 "s"(a.ws_all[3]), "s"(a.ws_all[4]), "s"(a.ws_all[5]), "s"(a.ws_all[6]), "s"(a.ws_all[7])); SB0
#define KLOCALS const int rank = a.rank; char* myws = a.myws; unsigned* tmo = (unsigned*)(myws + WS_TMO); \
                const unsigned* cntp = (const unsigned*)(myws + WS_CNT) + b
    if (!JMP) { KARGS; }

    if (__builtin_expect(wid < 8, 1)) {
        if (JMP) { KARGS; }
        STAMP(10);
        KLOCALS;
        unsigned call;
        {
            const int prg = (b >> 1) & 3, pks = b >> 3, p = (b & 1) * 8192 + tid * 16;
            const int row = p >> 7, chunk = ((p >> 4) & 7) ^ ((row >> 1) & 7);
            const signed char* src = a.x + (size_t)(prg * 128 + row) * KD + pks * 128 + chunk * 16;
            if (pks >= NFIRST / 2) __builtin_amdgcn_s_sleep(LATE);
            u32x4 v;
            asm volatile("global_load_dwordx4 %0, %2, off\n\tglobal_load_dword %1, %3, %4 sc1"
                         : "=&v"(v), "=&v"(call) : "v"(src), "v"(0), "s"(cntp) : "memory");
            unsigned wlo = 0, whi = 0;
#pragma unroll
            for (int q = 0; q < NR; ++q)
                asm volatile("v_writelane_b32 %0, %2, %4\n\tv_writelane_b32 %1, %3, %4" : "+v"(wlo), "+v"(whi)
                             : "s"((unsigned)(size_t)a.ws_all[q]), "s"((unsigned)((size_t)a.ws_all[q] >> 32)), "n"(q));
            bar();
            STAMP(8);
            asm volatile("s_waitcnt vmcnt(0)" : "+v"(v), "+v"(call) :: "memory");
            STAMP(11);
            call = __builtin_amdgcn_readfirstlane(call) + 1u;
            char* dst = myws + WS_X + (call & 1u) * X_PAR + prg * X_RG + pks * X_KS + p;
            asm volatile("global_store_dwordx4 %0, %1, off sc0 sc1\n\ts_nop 1" :: "v"(dst), "v"(v) : "memory");
            char* flag = (char*)(((size_t)whi << 32) | wlo) + WS_FLAG + (size_t)((rank * 4 + prg) * 2 + (pks >= NFIRST / 2)) * 128;
            asm volatile("s_waitcnt vmcnt(0)" : "+v"(flag) :: "memory");
            STAMP(12);
            unsigned arrived = 0;
            if (lane == 0) arrived = __hip_atomic_fetch_add((LAS unsigned*)(lds + LDS_CTL + 8), 1u, __ATOMIC_RELAXED, __HIP_MEMORY_SCOPE_WORKGROUP);
            if (__builtin_amdgcn_readfirstlane(arrived) == 7u && lane < NR)
                asm volatile("global_atomic_add %0, %1, off sc1" :: "v"(flag), "v"(1) : "memory");
        }
        STAMP(1);
        const int wm = wid >> 2, wn = wid & 3;
        const int q8 = (lane & 15) >> 1;
        unsigned aoff[2], boff[4];
        aoff[0] = (wm * 64 + (lane & 15)) * 128 + (((lane >> 4) ^ q8) << 4); aoff[1] = aoff[0] ^ 64u;
#pragma unroll
        for (int j = 0; j < 4; ++j)
            boff[j] = IMG + ((lane >> 4) * 16 + q8) * 256 + (((wn * 4 + j) ^ (q8 | (((lane >> 4) & 1) << 3))) << 4) + (lane & 1) * 8;
        i32x4 acc[4][4];
#pragma unroll
        for (int i = 0; i < 4; ++i)
#pragma unroll
            for (int j = 0; j < 4; ++j) acc[i][j] = (i32x4){0, 0, 0, 0};

        auto rdB1 = [&](Frags& f, unsigned so, int h, int j) INL {
            const unsigned ba = boff[j] + so + h * 16384;
            const i32x2 lo = __builtin_amdgcn_ds_read_tr8_b64_v2i32((LAS i32x2*)(lds + ba));
            const i32x2 hi = __builtin_amdgcn_ds_read_tr8_b64_v2i32((LAS i32x2*)(lds + ba + 2048));
            f.b[j] = (i32x4){lo.x, lo.y, hi.x, hi.y};
        };
        auto rdA1 = [&](Frags& f, unsigned so, int h, int i) INL { f.a[i] = *(const LAS i32x4*)(lds + aoff[h] + so + i * 2048); };
        auto rd = [&](Frags& f, unsigned so, int h) INL {
#pragma unroll
            for (int t = 0; t < 4; ++t) { rdB1(f, so, h, t); rdA1(f, so, h, t); }
        };
        auto rdB = [&](Frags& f, unsigned so, int h) INL {
#pragma unroll
            for (int t = 0; t < 4; ++t) rdB1(f, so, h, t);
        };
        auto rdA = [&](Frags& f, unsigned so, int h, int half) INL { rdA1(f, so, h, 2 * half); rdA1(f, so, h, 2 * half + 1); };
        auto mmr = [&](const Frags& f, int i) INL {
#pragma unroll
            for (int j = 0; j < 4; ++j) acc[i][j] = __builtin_amdgcn_mfma_i32_16x16x64_i8(f.a[i], f.b[j], acc[i][j], 0, 0, 0);
        };
        auto mmh = [&](const Frags& f, int half) INL { mmr(f, 2 * half); mmr(f, 2 * half + 1); };
        auto mm = [&](const Frags& f) INL { mmh(f, 0); mmh(f, 1); };
        auto sched = [&]() INL {
#pragma unroll
            for (int i = 0; i < 12; ++i) { __builtin_amdgcn_sched_group_barrier(0x100, 1, 0); __builtin_amdgcn_sched_group_barrier(0x8, 1, 0); }
            __builtin_amdgcn_sched_group_barrier(0x8, 4, 0);
        };
        float s0 = 0.f, s1 = 0.f;
        if (!SCL) asm volatile("s_load_dword %0, %2, 0x0\n\ts_load_dword %1, %3, 0x0\n\ts_waitcnt lgkmcnt(0)"
                               : "=&s"(s0), "=&s"(s1) : "s"(a.sx), "s"(a.sw) : "memory");
        bar();
        if (tid == 0) __hip_atomic_store((unsigned*)(myws + WS_CNT) + b, call, __ATOMIC_RELAXED, __HIP_MEMORY_SCOPE_AGENT);
        const unsigned slot = __builtin_amdgcn_readfirstlane(ctl[1]);
        const int peer = (a.xmap >> (4 * (slot >> 5))) & 7, rg = slot & 3, cg = (slot >> 2) & 7;
        const char* ob = (const char*)(a.out + (size_t)(peer * MS + rg * 128 + wm * 64) * NS + cg * 256 + wn * 64);
        if (SCL) asm volatile("s_load_dword %0, %2, 0x0\n\ts_load_dword %1, %3, 0x0\n\ts_waitcnt lgkmcnt(0)"
                              : "=&s"(s0), "=&s"(s1) : "s"(a.sx), "s"(a.sw) : "memory");
        float scale = s0 * s1;
        if (TW2 && wid == 2) {
            unsigned d0, d1;
            asm volatile("s_load_dword %0, %2, 0x0\n\ts_load_dword %1, %3, 0x0\n\ts_waitcnt lgkmcnt(0)"
                         : "=&s"(d0), "=&s"(d1) : "s"((const char*)a.w + ((size_t)(slot & 31) << 21)), "s"(ob + (size_t)(cg & 1) * 127 * NS * 4) : "memory");
        }
        if (wid == 1) {
            __builtin_amdgcn_s_sleep(20);
            if (wait_count(myws + WS_FLAG + ((peer * 4 + rg) * 2 + 1) * 128, lane, call * (64 - NFIRST), call, tmo)) ctl[0] = 1u;
            asm volatile("s_waitcnt vmcnt(0) lgkmcnt(0)" ::: "memory");
        }
        Frags f0, f1;
        unsigned sb = 0;
        bar();
        STAMP(3);
        rd(f0, sb, 0);
        for (int s = 0; s < KSTEPS - 4; ++s) {
            SB0; rd(f1, sb, 1); mm(f0); sched(); SB0;
            asm volatile("s_waitcnt lgkmcnt(0)" ::: "memory");
            bar();
            sb = (sb == (D - 1) * STG) ? 0u : sb + STG;
            rd(f0, sb, 0); mm(f1); sched();
        }
        unsigned S28 = (28 % D) * STG, S29 = (29 % D) * STG, S30 = (30 % D) * STG, S31 = (31 % D) * STG;
        asm volatile("" : "+s"(S28), "+s"(S29), "+s"(S30), "+s"(S31));
        if (__builtin_amdgcn_readfirstlane(ctl[0]) != 0u) scale = __builtin_nanf("");
        asm volatile("" : "+v"(scale));
        SB0; rd(f1, S28, 1); mm(f0); sched(); SB0;
        asm volatile("s_waitcnt lgkmcnt(0)" ::: "memory");
        bar();
        STAMP(5);
        rdA(f0, S29, 0, 0); rdB(f0, S29, 0); SB0; mm(f1); SB0;
        rdA(f1, S29, 1, 0); rdB(f1, S29, 1); SB0; mmh(f0, 0); SB0;
        bar();
        rdA(f0, S30, 0, 0); rdB(f0, S30, 0); SB0; mmh(f1, 0); SB0;
        rdA(f1, S30, 1, 0); rdB(f1, S30, 1); SB0; mmh(f0, 0); SB0;
        bar();
        rdA(f0, S31, 0, 0); rdB(f0, S31, 0); SB0; mmh(f1, 0); SB0;
        rdA(f1, S31, 1, 0); rdB(f1, S31, 1); SB0; mmh(f0, 0); SB0;
        rdB(f0, S29, 0); SB0; mmh(f1, 0); SB0;
        rdB(f1, S29, 1); SB0;
        unsigned ov;
        asm volatile("v_mbcnt_lo_u32_b32 %0, -1, 0\n\tv_mbcnt_hi_u32_b32 %0, -1, %0" : "=v"(ov));
        ov = (ov >> 5) * 8 * NS * 4 + (ov & 31) * 4;
        auto st = [&](int n) INL {
            const int i = n >> 3, jp = (n >> 2) & 1, e = n & 3;
            const auto sw = __builtin_amdgcn_permlane16_swap((unsigned)acc[i][2 * jp][e], (unsigned)acc[i][2 * jp + 1][e], false, false);
            const float y0 = __builtin_elementwise_maximum((float)(int)sw[0] * scale, 0.0f);
            const float y1 = __builtin_elementwise_maximum((float)(int)sw[1] * scale, 0.0f);
            const char* rb = ob + (size_t)(i * 16 + e) * NS * 4 + jp * 128;
            asm volatile("global_store_dword %0, %1, %3 sc1\n\tglobal_store_dword %0, %2, %4 sc1"
                         :: "v"(ov), "v"(y0), "v"(y1), "s"(rb), "s"(rb + 4 * NS * 4) : "memory");
        };
        STAMP(4);
#pragma unroll
        for (int n = 0; n < 9; ++n) { st(n); SB0; }
        bar();
        rdA(f0, S29, 0, 1); rdA(f1, S29, 1, 1); SB0; mmh(f0, 1); SB0; rdB(f0, S30, 0); SB0; mmh(f1, 1); SB0; rdB(f1, S30, 1); SB0;
#pragma unroll
        for (int n = 9; n < 13; ++n) { st(n); SB0; }
        bar();
        rdA(f0, S30, 0, 1); rdA(f1, S30, 1, 1); SB0; mmh(f0, 1); SB0; rdB(f0, S31, 0); SB0; mmh(f1, 1); SB0; rdB(f1, S31, 1); SB0;
#pragma unroll
        for (int n = 13; n < 16; ++n) { st(n); SB0; }
        bar();
        rdA(f0, S31, 0, 1); rdA(f1, S31, 1, 1); SB0; mmh(f0, 1); mmh(f1, 1); SB0;
        STAMP(7);
#pragma unroll
        for (int n = 16; n < 32; ++n) { st(n); SB0; }
    } else {
        if (JMP) { KARGS; }
        KLOCALS;
        const int lw = wid - 8;
        const unsigned xcc = (unsigned)__builtin_amdgcn_s_getreg(20 | (3 << 11)) & 7u;
        unsigned* claim = (unsigned*)(myws + WS_CLAIM);
        unsigned slot = xcc * 32u + (unsigned)(b >> 3);
        unsigned call, old = 0;
        asm volatile("global_load_dword %0, %1, %2 sc1" : "=v"(call) : "v"(0), "s"(cntp) : "memory");
        if (lw == 0 && lane == 0)
            asm volatile("global_atomic_add %0, %1, %2, %3 sc0" : "=v"(old) : "v"(0), "v"(1), "s"(claim + slot * 32) : "memory");
        unsigned touch = 0;
        if (TW1 && lw == 3 && (b >> 3) >= TW0 && (b >> 3) < TW0 + TW1) {
            const char* pw = a.ws_all[(a.xmap >> (4 * xcc)) & 7];
            const char* t = lane < 12 ? myws : lane < 16 ? pw : (const char*)a.w;
            unsigned lo = (unsigned)(size_t)t, hi = (unsigned)((size_t)t >> 32);
#pragma unroll
            for (int q = 0; q < NR; ++q)
                asm volatile("v_writelane_b32 %0, %2, %4\n\tv_writelane_b32 %1, %3, %4" : "+v"(lo), "+v"(hi)
                             : "s"((unsigned)(size_t)a.ws_all[q]), "s"((unsigned)((size_t)a.ws_all[q] >> 32)), "n"(q));
            const unsigned o0 = WS_PAD + (rank * 32 + (b & 31)) * 128, o1 = WS_X + ((lane & 3) * 8 + 8) * X_KS - 128, o2 = (lane & 3) << 21;
            t = (const char*)(((size_t)hi << 32) | lo) + (lane < 8 ? o0 : lane < 16 ? o1 : o2);
            if (TDEL) __builtin_amdgcn_s_sleep(TDEL);
            if (lane < 32 && ((TM1 >> lane) & 1u)) asm volatile("global_load_dword %0, %1, off sc0 sc1" : "=v"(touch) : "v"(t) : "memory");
            if (TM2) {
                wait_vm<1>();
                if (lane < 32 && ((TM2 >> lane) & 1u)) asm volatile("global_load_dword %0, %1, off sc0 sc1" : "+v"(touch) : "v"(t) : "memory");
            }
        }
        const unsigned lds0 = (unsigned)(size_t)lds, ldsA = lds0 + lw * 4096, ldsB = lds0 + IMG + lw * 8192;
        const unsigned voffA = lane * 16;
        unsigned voff[4];
#pragma unroll
        for (int i = 0; i < 4; ++i) { const int k = (i & 1) * 4 + (lane >> 4);
            voff[i] = k * ND + (((lane & 15) ^ (k | ((i >> 1) << 3))) << 4); }
        const char* gW = (const char*)a.w + (size_t)rank * NS + (size_t)(lw * 32) * ND;
        auto ld1 = [&](const char* g, unsigned l, unsigned v) INL {
            unsigned keep;
            asm volatile("s_mov_b32 %0, m0\n\ts_mov_b32 m0, %3\n\ts_nop 0\n\tglobal_load_lds_dwordx4 %1, %2\n\ts_mov_b32 m0, %0"
                         : "=&s"(keep) : "v"(v), "s"(g), "s"(l) : "memory");
        };
        if (lw == 0 && lane < 4) ctl[lane] = 0u;
        asm volatile("s_waitcnt lgkmcnt(0)" ::: "memory");
        bar();
        asm volatile("s_waitcnt vmcnt(0)" : "+v"(call), "+v"(old), "+v"(touch), "+v"(cpf) :: "memory");
        call = __builtin_amdgcn_readfirstlane(call) + 1u;
        STAMP(9);
        if (lw == 0) {
            unsigned lost = 0;
            if (lane == 0) {
#pragma nounroll
                for (unsigned tries = 0; old != call - 1u; ++tries) {
                    __hip_atomic_fetch_sub(claim + slot * 32, 1u, __ATOMIC_RELAXED, __HIP_MEMORY_SCOPE_AGENT);
                    if (tries >= GRID) { lost = 1u; break; }
                    slot = (slot + 1u) & (GRID - 1);
                    old = __hip_atomic_fetch_add(claim + slot * 32, 1u, __ATOMIC_RELAXED, __HIP_MEMORY_SCOPE_AGENT);
                }
            }
            slot = __builtin_amdgcn_readfirstlane(slot);
            lost = __builtin_amdgcn_readfirstlane(lost);
            const int pr = (a.xmap >> (4 * (slot >> 5))) & 7;
            const char* g0 = a.ws_all[pr] + WS_X + (call & 1u) * X_PAR + (slot & 3) * X_RG;
            asm volatile("" :: "s"(g0));
            ctl[1] = slot;
            if (wait_count(myws + WS_FLAG + (pr * 4 + (slot & 3)) * 2 * 128, lane, call * NFIRST, call, tmo)) lost = 1u;
            STAMP(2);
            if (lost) ctl[0] = 1u;
            else {
#pragma unroll
                for (int i = 0; i < 8; ++i) ld1(g0 + (i >> 1) * 4096 + (i & 1) * 1024, lds0 + (i >> 1) * 4096 + (i & 1) * 1024, voffA);
            }
            asm volatile("s_waitcnt lgkmcnt(0)" ::: "memory");
        }
        bar();
        const bool bad = __builtin_amdgcn_readfirstlane(ctl[0]) != 0u;
        slot = __builtin_amdgcn_readfirstlane(ctl[1]);
        const int peer = (a.xmap >> (4 * (slot >> 5))) & 7, rg = slot & 3, cg = (slot >> 2) & 7;
        const char* gB = gW + cg * 256;
        const char* gR = a.ws_all[bad ? rank : peer] + WS_X + (call & 1u) * X_PAR + rg * X_RG;
        auto ldA = [&](int s, unsigned so, int i) INL { ld1(gR + s * X_KS + lw * 4096 + i * 1024, ldsA + so + i * 1024, voffA); };
        auto ldB = [&](int s, unsigned so, int i) INL {
            const char* g = gB + (size_t)(s * 128 + i * 8) * ND;
            ld1(g, ldsB + so + i * 2048, voff[i & 2]);
            ld1(g, ldsB + so + i * 2048 + 1024, voff[(i & 2) + 1]);
        };
        auto A = [&](int n) INL { ldA(n >> 2, (n >> 2) * STG, n & 3); };
        auto B = [&](int n) INL { ldB(n >> 2, (n >> 2) * STG, n & 3); };
        B(0); A(2); B(1); A(3); B(2); A(4); B(3);
        A(5); B(4); A(6); B(5); B(6); A(7); B(7);
        A(8); B(8); A(9); B(9); B(10); A(10); B(11); A(11);
        wait_vm<23>();
        bar();
        if (__builtin_amdgcn_readfirstlane(ctl[0]) != 0u) gR = myws + WS_X;
        const bool lead = peer != rank && rg == 0 && lw < 2;
        auto early = [&](int s) INL { if (lead) ld1(gR + s * X_KS + (cg * 2 + lw) * 1024, lds0 + LDS_SCR + lw * 1024, voffA); };
        wait_vm<12>();
        unsigned sn = 0;
        for (int s = 1; s < KSTEPS - 5; ++s) {
            bar();
            early(s + 3);
#pragma unroll
            for (int i = 0; i < 4; ++i) { ldB(s + 2, sn, i); ldA(s + 2, sn, i); }
            sn = (sn == (D - 1) * STG) ? 0u : sn + STG;
            if (lead) wait_vm<13>(); else wait_vm<12>();
        }
        const unsigned hoff = (lw >> 1) * 8192 + (lw & 1) * 2048;
        auto ldH = [&](int s, int half) INL {
            ld1(gR + s * X_KS + hoff + half * 4096, lds0 + (s % D) * STG + hoff + half * 4096, voffA);
            ld1(gR + s * X_KS + hoff + half * 4096 + 1024, lds0 + (s % D) * STG + hoff + half * 4096 + 1024, voffA);
        };
        auto ldU = [&](int s) INL {
            ldH(s, 0);
#pragma unroll
            for (int i = 0; i < 4; ++i) ldB(s, (s % D) * STG, i);
        };
        bar();
        early(30); ldU(29);
        if (lead) wait_vm<11>(); else wait_vm<10>();
        bar();
        early(31); ldU(30);
        if (lead) wait_vm<11>(); else wait_vm<10>();
        bar();
        ldU(31);
        wait_vm<10>();
        bar();
        ldH(29, 1); ldH(30, 1); ldH(31, 1);
        wait_vm<6>();
        bar();
        wait_vm<4>();
        bar();
        wait_vm<2>();
        bar();
        if (CPF) asm volatile(".Lgg_end:");
        wait_vm<0>();
        bar();
    }
}

}

extern "C" void kernel_launch(int rank, int nranks, ncclComm_t comm,
                              hipStream_t stream,
                              void* const* d_in, const int* in_sizes, int n_in,
                              void* d_out, int out_size,
                              const int* in_dtypes, int out_dtype,
                              void* const* d_out_all,
                              void* const* d_ws_all, size_t ws_bytes) {
    if (nranks != NR || (unsigned)rank >= (unsigned)NR || n_in != 4 || ws_bytes < WS_NEED ||
        in_sizes[0] != MS * KD || in_sizes[1] != KD * ND || out_size != NR * MS * NS) {
        fprintf(stderr, "kernel_launch: unexpected shapes\n");
        return;
    }
    Args a;
    a.x = (const signed char*)d_in[0]; a.w = (const signed char*)d_in[1];
    a.sx = (const float*)d_in[2]; a.sw = (const float*)d_in[3];
    a.out = (float*)d_out; a.rank = rank; a.myws = (char*)d_ws_all[rank]; a.stamps = nullptr;
    static const unsigned kXmap[NR] = {0x74513206u, 0x26305417u, 0x17546023u, 0x47150632u,
                                       0x36027145u, 0x60231754u, 0x51472360u, 0x02364571u};
    a.xmap = kXmap[rank];
    for (int p = 0; p < NR; ++p) a.ws_all[p] = (char*)d_ws_all[p];
    hipLaunchKernelGGL(gemm_gather_kernel, dim3(GRID), dim3(BLOCK), 0, stream, a);
    HIP_CHECK(hipGetLastError());
}
```

```cpp
#include <hip/hip_runtime.h>
#include <rccl/rccl.h>
#include <cstdio>

#define HIP_CHECK(expr) do { hipError_t _e = (expr); if (_e != hipSuccess) { \
    fprintf(stderr, "HIP error %s at %s:%d\n", hipGetErrorString(_e), __FILE__, __LINE__); \
    return; } } while (0)

namespace {

#define LAS __attribute__((address_space(3)))
typedef unsigned u32x4 __attribute__((ext_vector_type(4)));
typedef int i32x4 __attribute__((ext_vector_type(4)));
typedef int i32x2 __attribute__((ext_vector_type(2)));

constexpr int NR = 8;
constexpr int KD = 4096, ND = 16384;
constexpr int MS = 512, NS = 2048;
constexpr int GRID = 256, BLOCK = 768;
constexpr int IMG = 16384;
constexpr int STG = 3 * IMG;
constexpr int KSTEPS = KD / 128;
constexpr int D = 3;
constexpr int LDS_CTL = D * STG;
constexpr int LDS_SCR = LDS_CTL + 1024;
constexpr int LDS_BYTES = LDS_SCR + 2048;
constexpr int NFIRST = 6;
constexpr int NP = 2, PK0 = 5;

constexpr size_t WS_CNT = 0;
constexpr size_t WS_TMO = 4096;
constexpr size_t WS_FLAG = 8192;
constexpr size_t WS_CLAIM = 32768;
constexpr size_t WS_PAD = 65536;
constexpr size_t WS_X = 131072;
constexpr size_t X_KS = 8 * IMG + 256, X_PAR = 4 * IMG, X_RG = IMG;
constexpr size_t WS_R = WS_X + KSTEPS * X_KS;
constexpr size_t WS_NEED = WS_R + (size_t)2 * NR * NP * IMG;
static_assert(WS_FLAG + (size_t)NR * 8 * 128 <= WS_CLAIM && WS_CLAIM + (size_t)GRID * 128 <= WS_PAD, "layout");
constexpr unsigned long long DEADLINE_TICKS = 50000000ull;

struct Args {
    const signed char* x; char* myws; const float* sx; const float* sw;
    const signed char* w; float* out; int rank; unsigned xmap;
    unsigned long long* stamps;
    char* ws_all[NR];
};

template <int N> __device__ __forceinline__ void wait_vm() { asm volatile("s_waitcnt vmcnt(%0)" :: "n"(N) : "memory"); }
__device__ __forceinline__ void bar() { asm volatile("" ::: "memory"); __builtin_amdgcn_s_barrier(); asm volatile("" ::: "memory"); }
#define SB0 __builtin_amdgcn_sched_barrier(0)
#define INL __attribute__((always_inline))
#define STAMP(i) do {} while (0)

__device__ __forceinline__ bool wait_count(const char* f, int lane, unsigned want, unsigned call, unsigned* tmo) {
    const unsigned long long t0 = __builtin_amdgcn_s_memrealtime();
    for (unsigned it = 1;; ++it) {
        unsigned v = 0;
        if (lane == 0) asm volatile("global_atomic_add %0, %1, %1, %2 sc0 sc1\n\ts_waitcnt vmcnt(0)" : "=&v"(v) : "v"(0), "s"(f) : "memory");
        if (__builtin_amdgcn_readfirstlane(v) >= want) return false;
        if ((it & 63u) == 0u) {
            if (__builtin_amdgcn_readfirstlane(__hip_atomic_load(tmo, __ATOMIC_RELAXED, __HIP_MEMORY_SCOPE_AGENT)) == call
                || __builtin_amdgcn_s_memrealtime() - t0 > DEADLINE_TICKS) {
                __hip_atomic_store(tmo, call, __ATOMIC_RELAXED, __HIP_MEMORY_SCOPE_AGENT);
                return true;
            }
        }
        if (it > 64u) __builtin_amdgcn_s_sleep(4);
    }
}

static_assert(PK0 >= 3 && PK0 + 7 * NP <= 29 && NP <= 3, "pushed images belong to the main loop, one per row group and peer");

struct Frags { i32x4 a[4], b[4]; };

__global__ __launch_bounds__(BLOCK, 3) void gemm_gather_kernel(Args a) {
    __shared__ __attribute__((aligned(16))) char smem[LDS_BYTES];
    LAS char* lds = (LAS char*)smem;
    const int tid = threadIdx.x, lane = tid & 63, wid = __builtin_amdgcn_readfirstlane(tid >> 6);
    const int b = blockIdx.x;
    STAMP(0);
    volatile LAS unsigned* ctl = (volatile LAS unsigned*)(lds + LDS_CTL);
#define KARGS asm volatile("" :: "s"(a.x), "s"(a.myws), "s"(a.w), "s"(a.rank), "s"(a.xmap), "s"(a.ws_all[0]), "s"(a.ws_all[1]), "s"(a.ws_all[2]), \
                 "s"(a.ws_all[3]), "s"(a.ws_all[4]), "s"(a.ws_all[5]), "s"(a.ws_all[6]), "s"(a.ws_all[7])); SB0
#define KLOCALS const int rank = a.rank; char* myws = a.myws; unsigned* tmo = (unsigned*)(myws + WS_TMO); \
                const unsigned* cntp = (const unsigned*)(myws + WS_CNT) + b

    if (__builtin_expect(wid < 8, 1)) {
        KARGS;
        STAMP(10);
        KLOCALS;
        unsigned call;
        {
            const int prg = (b >> 1) & 3, pks = b >> 3, p = (b & 1) * 8192 + tid * 16;
            const int row = p >> 7, chunk = ((p >> 4) & 7) ^ ((row >> 1) & 7);
            const signed char* src = a.x + (size_t)(prg * 128 + row) * KD + pks * 128 + chunk * 16;
            const int pj = pks - PK0;
            const bool push = pj >= 0 && pj < 7 * NP && prg == 1 + pj % 3;
            if (pks >= NFIRST / 2 && !(push && pj < 7)) __builtin_amdgcn_s_sleep(8);
            u32x4 v;
            asm volatile("global_load_dwordx4 %0, %2, off\n\tglobal_load_dword %1, %3, %4 sc1"
                         : "=&v"(v), "=&v"(call) : "v"(src), "v"(0), "s"(cntp) : "memory");
            unsigned wlo = 0, whi = 0;
#pragma unroll
            for (int q = 0; q < NR; ++q)
                asm volatile("v_writelane_b32 %0, %2, %4\n\tv_writelane_b32 %1, %3, %4" : "+v"(wlo), "+v"(whi)
                             : "s"((unsigned)(size_t)a.ws_all[q]), "s"((unsigned)((size_t)a.ws_all[q] >> 32)), "n"(q));
            bar();
            STAMP(8);
            asm volatile("s_waitcnt vmcnt(0)" : "+v"(v), "+v"(call) :: "memory");
            STAMP(11);
            call = __builtin_amdgcn_readfirstlane(call) + 1u;
            char* dst = myws + WS_X + (call & 1u) * X_PAR + prg * X_RG + pks * X_KS + p;
            asm volatile("global_store_dwordx4 %0, %1, off sc0 sc1\n\ts_nop 1" :: "v"(dst), "v"(v) : "memory");
            if (push) {
                char* dst2 = a.ws_all[(rank + 1 + pj % 7) & 7] + WS_R + (size_t)(((call & 1u) * NR + rank) * NP + pj / 7) * IMG + p;
                if (pj >= 7) wait_vm<0>();
                asm volatile("global_store_dwordx4 %0, %1, off sc0 sc1\n\ts_nop 1" :: "v"(dst2), "v"(v) : "memory");
            }
            char* flag = (char*)(((size_t)whi << 32) | wlo) + WS_FLAG + (size_t)((rank * 4 + prg) * 2 + (pks >= NFIRST / 2)) * 128;
            asm volatile("s_waitcnt vmcnt(0)" : "+v"(flag) :: "memory");
            STAMP(12);
            unsigned arrived = 0;
            if (lane == 0) arrived = __hip_atomic_fetch_add((LAS unsigned*)(lds + LDS_CTL + 8), 1u, __ATOMIC_RELAXED, __HIP_MEMORY_SCOPE_WORKGROUP);
            if (__builtin_amdgcn_readfirstlane(arrived) == 7u && lane < NR)
                asm volatile("global_atomic_add %0, %1, off sc1" :: "v"(flag), "v"(1) : "memory");
        }
        STAMP(1);
        const int wm = wid >> 2, wn = wid & 3;
        const int q8 = (lane & 15) >> 1;
        unsigned aoff[2], boff[4];
        aoff[0] = (wm * 64 + (lane & 15)) * 128 + (((lane >> 4) ^ q8) << 4); aoff[1] = aoff[0] ^ 64u;
#pragma unroll
        for (int j = 0; j < 4; ++j)
            boff[j] = IMG + ((lane >> 4) * 16 + q8) * 256 + (((wn * 4 + j) ^ (q8 | (((lane >> 4) & 1) << 3))) << 4) + (lane & 1) * 8;
        i32x4 acc[4][4];
#pragma unroll
        for (int i = 0; i < 4; ++i)
#pragma unroll
            for (int j = 0; j < 4; ++j) acc[i][j] = (i32x4){0, 0, 0, 0};

        auto rdB1 = [&](Frags& f, unsigned so, int h, int j) INL {
            const unsigned ba = boff[j] + so + h * 16384;
            const i32x2 lo = __builtin_amdgcn_ds_read_tr8_b64_v2i32((LAS i32x2*)(lds + ba));
            const i32x2 hi = __builtin_amdgcn_ds_read_tr8_b64_v2i32((LAS i32x2*)(lds + ba + 2048));
            f.b[j] = (i32x4){lo.x, lo.y, hi.x, hi.y};
        };
        auto rdA1 = [&](Frags& f, unsigned so, int h, int i) INL { f.a[i] = *(const LAS i32x4*)(lds + aoff[h] + so + i * 2048); };
        auto rd = [&](Frags& f, unsigned so, int h) INL {
#pragma unroll
            for (int t = 0; t < 4; ++t) { rdB1(f, so, h, t); rdA1(f, so, h, t); }
        };
        auto rdB = [&](Frags& f, unsigned so, int h) INL {
#pragma unroll
            for (int t = 0; t < 4; ++t) rdB1(f, so, h, t);
        };
        auto rdA = [&](Frags& f, unsigned so, int h, int half) INL { rdA1(f, so, h, 2 * half); rdA1(f, so, h, 2 * half + 1); };
        auto mmr = [&](const Frags& f, int i) INL {
#pragma unroll
            for (int j = 0; j < 4; ++j) acc[i][j] = __builtin_amdgcn_mfma_i32_16x16x64_i8(f.a[i], f.b[j], acc[i][j], 0, 0, 0);
        };
        auto mmh = [&](const Frags& f, int half) INL { mmr(f, 2 * half); mmr(f, 2 * half + 1); };
        auto mm = [&](const Frags& f) INL { mmh(f, 0); mmh(f, 1); };
        auto sched = [&]() INL {
#pragma unroll
            for (int i = 0; i < 12; ++i) { __builtin_amdgcn_sched_group_barrier(0x100, 1, 0); __builtin_amdgcn_sched_group_barrier(0x8, 1, 0); }
            __builtin_amdgcn_sched_group_barrier(0x8, 4, 0);
        };
        bar();
        const unsigned slot = __builtin_amdgcn_readfirstlane(ctl[1]);
        const int peer = (a.xmap >> (4 * (slot >> 5))) & 7, rg = slot & 3, cg = (slot >> 2) & 7;
        const char* ob = (const char*)(a.out + (size_t)(peer * MS + rg * 128 + wm * 64) * NS + cg * 256 + wn * 64);
        float s0, s1;
        asm volatile("s_load_dword %0, %2, 0x0\n\ts_load_dword %1, %3, 0x0\n\ts_waitcnt lgkmcnt(0)"
                     : "=&s"(s0), "=&s"(s1) : "s"(a.sx), "s"(a.sw) : "memory");
        float scale = s0 * s1;
        if (wid == 2)
            asm volatile("s_load_dword %0, %2, 0x0\n\ts_load_dword %1, %3, 0x0\n\ts_waitcnt lgkmcnt(0)"
                         : "=&s"(s0), "=&s"(s1) : "s"((const char*)a.w + ((size_t)(slot & 31) << 21)), "s"(ob + (size_t)(cg & 1) * 127 * NS * 4) : "memory");
        if (wid == 1) {
            __builtin_amdgcn_s_sleep(20);
            if (wait_count(myws + WS_FLAG + ((peer * 4 + rg) * 2 + 1) * 128, lane, call * (64 - NFIRST), call, tmo)) ctl[0] = 1u;
            asm volatile("s_waitcnt vmcnt(0) lgkmcnt(0)" ::: "memory");
        }
        Frags f0, f1;
        unsigned sb = 0;
        bar();
        if (tid == 0) __hip_atomic_store((unsigned*)(myws + WS_CNT) + b, call, __ATOMIC_RELAXED, __HIP_MEMORY_SCOPE_AGENT);
        STAMP(3);
        rd(f0, sb, 0);
        for (int s = 0; s < KSTEPS - 4; ++s) {
            SB0; rd(f1, sb, 1); mm(f0); sched(); SB0;
            asm volatile("s_waitcnt lgkmcnt(0)" ::: "memory");
            bar();
            sb = (sb == (D - 1) * STG) ? 0u : sb + STG;
            rd(f0, sb, 0); mm(f1); sched();
        }
        unsigned S28 = (28 % D) * STG, S29 = (29 % D) * STG, S30 = (30 % D) * STG, S31 = (31 % D) * STG;
        asm volatile("" : "+s"(S28), "+s"(S29), "+s"(S30), "+s"(S31));
        if (__builtin_amdgcn_readfirstlane(ctl[0]) != 0u) scale = __builtin_nanf("");
        asm volatile("" : "+v"(scale));
        SB0; rd(f1, S28, 1); mm(f0); sched(); SB0;
        asm volatile("s_waitcnt lgkmcnt(0)" ::: "memory");
        bar();
        STAMP(5);
        rdA(f0, S29, 0, 0); rdB(f0, S29, 0); SB0; mm(f1); SB0;
        rdA(f1, S29, 1, 0); rdB(f1, S29, 1); SB0; mmh(f0, 0); SB0;
        bar();
        rdA(f0, S30, 0, 0); rdB(f0, S30, 0); SB0; mmh(f1, 0); SB0;
        rdA(f1, S30, 1, 0); rdB(f1, S30, 1); SB0; mmh(f0, 0); SB0;
        bar();
        rdA(f0, S31, 0, 0); rdB(f0, S31, 0); SB0; mmh(f1, 0); SB0;
        rdA(f1, S31, 1, 0); rdB(f1, S31, 1); SB0; mmh(f0, 0); SB0;
        rdB(f0, S29, 0); SB0; mmh(f1, 0); SB0;
        rdB(f1, S29, 1); SB0;
        unsigned ov;
        asm volatile("v_mbcnt_lo_u32_b32 %0, -1, 0\n\tv_mbcnt_hi_u32_b32 %0, -1, %0" : "=v"(ov));
        ov = (ov >> 5) * 8 * NS * 4 + (ov & 31) * 4;
        auto st = [&](int n) INL {
            const int i = n >> 3, jp = (n >> 2) & 1, e = n & 3;
            const auto sw = __builtin_amdgcn_permlane16_swap((unsigned)acc[i][2 * jp][e], (unsigned)acc[i][2 * jp + 1][e], false, false);
            const float y0 = __builtin_elementwise_maximum((float)(int)sw[0] * scale, 0.0f);
            const float y1 = __builtin_elementwise_maximum((float)(int)sw[1] * scale, 0.0f);
            const char* rb = ob + (size_t)(i * 16 + e) * NS * 4 + jp * 128;
            asm volatile("global_store_dword %0, %1, %3 sc1\n\tglobal_store_dword %0, %2, %4 sc1"
                         :: "v"(ov), "v"(y0), "v"(y1), "s"(rb), "s"(rb + 4 * NS * 4) : "memory");
        };
        STAMP(4);
#pragma unroll
        for (int n = 0; n < 9; ++n) { st(n); SB0; }
        bar();
        rdA(f0, S29, 0, 1); rdA(f1, S29, 1, 1); SB0; mmh(f0, 1); SB0; rdB(f0, S30, 0); SB0; mmh(f1, 1); SB0; rdB(f1, S30, 1); SB0;
#pragma unroll
        for (int n = 9; n < 13; ++n) { st(n); SB0; }
        bar();
        rdA(f0, S30, 0, 1); rdA(f1, S30, 1, 1); SB0; mmh(f0, 1); SB0; rdB(f0, S31, 0); SB0; mmh(f1, 1); SB0; rdB(f1, S31, 1); SB0;
#pragma unroll
        for (int n = 13; n < 16; ++n) { st(n); SB0; }
        bar();
        rdA(f0, S31, 0, 1); rdA(f1, S31, 1, 1); SB0; mmh(f0, 1); mmh(f1, 1); SB0;
        STAMP(7);
#pragma unroll
        for (int n = 16; n < 32; ++n) { st(n); SB0; }
    } else {
        KARGS;
        KLOCALS;
        const int lw = wid - 8;
        const unsigned xcc = (unsigned)__builtin_amdgcn_s_getreg(20 | (3 << 11)) & 7u;
        unsigned* claim = (unsigned*)(myws + WS_CLAIM);
        unsigned slot = xcc * 32u + (unsigned)(b >> 3);
        unsigned call, old = 0;
        asm volatile("global_load_dword %0, %1, %2 sc1" : "=v"(call) : "v"(0), "s"(cntp) : "memory");
        if (lw == 0 && lane == 0)
            asm volatile("global_atomic_add %0, %1, %2, %3 sc0" : "+v"(old) : "v"(0), "v"(1), "s"(claim + slot * 32) : "memory");
        unsigned touch = 0;
        if (lw == 3 && (b >> 3) >= 3 && (b >> 3) < 5) {
            const char* pw = a.ws_all[(a.xmap >> (4 * xcc)) & 7];
            const char* t = lane < 12 ? myws : lane < 16 ? pw : (const char*)a.w;
            unsigned lo = (unsigned)(size_t)t, hi = (unsigned)((size_t)t >> 32);
#pragma unroll
            for (int q = 0; q < NR; ++q)
                asm volatile("v_writelane_b32 %0, %2, %4\n\tv_writelane_b32 %1, %3, %4" : "+v"(lo), "+v"(hi)
                             : "s"((unsigned)(size_t)a.ws_all[q]), "s"((unsigned)((size_t)a.ws_all[q] >> 32)), "n"(q));
            const unsigned o0 = WS_PAD + (rank * 32 + (b & 31)) * 128, o1 = WS_X + ((lane & 3) * 8 + 8) * X_KS - 128, o2 = (lane & 3) << 21;
            t = (const char*)(((size_t)hi << 32) | lo) + (lane < 8 ? o0 : lane < 16 ? o1 : o2);
            if (lane < 8 || lane == 16) asm volatile("global_load_dword %0, %1, off sc0 sc1" : "+v"(touch) : "v"(t) : "memory");
            wait_vm<1>();
            if (lane >= 8 && lane < 19 && lane != 16) asm volatile("global_load_dword %0, %1, off sc0 sc1" : "+v"(touch) : "v"(t) : "memory");
        }
        const unsigned lds0 = (unsigned)(size_t)lds, ldsA = lds0 + lw * 4096, ldsB = lds0 + IMG + lw * 8192;
        const unsigned voffA = lane * 16;
        unsigned voff[4];
#pragma unroll
        for (int i = 0; i < 4; ++i) { const int k = (i & 1) * 4 + (lane >> 4);
            voff[i] = k * ND + (((lane & 15) ^ (k | ((i >> 1) << 3))) << 4); }
        const char* gW = (const char*)a.w + (size_t)rank * NS + (size_t)(lw * 32) * ND;
        auto ld1 = [&](const char* g, unsigned l, unsigned v) INL {
            unsigned keep;
            asm volatile("s_mov_b32 %0, m0\n\ts_mov_b32 m0, %3\n\ts_nop 0\n\tglobal_load_lds_dwordx4 %1, %2\n\ts_mov_b32 m0, %0"
                         : "=&s"(keep) : "v"(v), "s"(g), "s"(l) : "memory");
        };
        if (lw == 0 && lane < 4) ctl[lane] = 0u;
        asm volatile("s_waitcnt lgkmcnt(0)" ::: "memory");
        bar();
        asm volatile("s_waitcnt vmcnt(0)" : "+v"(call), "+v"(old), "+v"(touch) :: "memory");
        call = __builtin_amdgcn_readfirstlane(call) + 1u;
        STAMP(9);
        if (lw == 0) {
            unsigned lost = 0;
            if (lane == 0) {
#pragma nounroll
                for (unsigned tries = 0; old != call - 1u; ++tries) {
                    __hip_atomic_fetch_sub(claim + slot * 32, 1u, __ATOMIC_RELAXED, __HIP_MEMORY_SCOPE_AGENT);
                    if (tries >= GRID) { lost = 1u; break; }
                    slot = (slot + 1u) & (GRID - 1);
                    old = __hip_atomic_fetch_add(claim + slot * 32, 1u, __ATOMIC_RELAXED, __HIP_MEMORY_SCOPE_AGENT);
                }
            }
            slot = __builtin_amdgcn_readfirstlane(slot);
            lost = __builtin_amdgcn_readfirstlane(lost);
            const int pr = (a.xmap >> (4 * (slot >> 5))) & 7;
            const char* g0 = a.ws_all[pr] + WS_X + (call & 1u) * X_PAR + (slot & 3) * X_RG;
            asm volatile("" :: "s"(g0));
            ctl[1] = slot;
            if (wait_count(myws + WS_FLAG + (pr * 4 + (slot & 3)) * 2 * 128, lane, call * NFIRST, call, tmo)) lost = 1u;
            STAMP(2);
            if (lost) ctl[0] = 1u;
            else {
#pragma unroll
                for (int i = 0; i < 8; ++i) ld1(g0 + (i >> 1) * 4096 + (i & 1) * 1024, lds0 + (i >> 1) * 4096 + (i & 1) * 1024, voffA);
            }
            asm volatile("s_waitcnt lgkmcnt(0)" ::: "memory");
        }
        bar();
        const bool bad = __builtin_amdgcn_readfirstlane(ctl[0]) != 0u;
        slot = __builtin_amdgcn_readfirstlane(ctl[1]);
        const int peer = (a.xmap >> (4 * (slot >> 5))) & 7, rg = slot & 3, cg = (slot >> 2) & 7;
        const char* gB = gW + cg * 256;
        const char* gR = a.ws_all[bad ? rank : peer] + WS_X + (call & 1u) * X_PAR + rg * X_RG;
        auto ldA = [&](int s, unsigned so, int i) INL { ld1(gR + s * X_KS + lw * 4096 + i * 1024, ldsA + so + i * 1024, voffA); };
        auto ldB = [&](int s, unsigned so, int i) INL {
            const char* g = gB + (size_t)(s * 128 + i * 8) * ND;
            ld1(g, ldsB + so + i * 2048, voff[i & 2]);
            ld1(g, ldsB + so + i * 2048 + 1024, voff[(i & 2) + 1]);
        };
        auto A = [&](int n) INL { ldA(n >> 2, (n >> 2) * STG, n & 3); };
        auto B = [&](int n) INL { ldB(n >> 2, (n >> 2) * STG, n & 3); };
        B(0); A(2); B(1); A(3); B(2); A(4); B(3);
        A(5); B(4); A(6); B(5); B(6); A(7); B(7);
        A(8); B(8); A(9); B(9); B(10); A(10); B(11); A(11);
        wait_vm<23>();
        bar();
        if (__builtin_amdgcn_readfirstlane(ctl[0]) != 0u) gR = myws + WS_X;
        int pk = -1; const char* gP = gR;
#pragma unroll
        for (int n = 0; n < NP; ++n) {
            const int j = ((rank - peer - 1) & 7) + 7 * n;
            if (peer != rank && rg == 1 + j % 3) { pk = PK0 + j; gP = myws + WS_R + (size_t)(((call & 1u) * NR + peer) * NP + n) * IMG; }
        }
        const bool lead = peer != rank && rg == 0 && lw < 2;
        auto early = [&](int s) INL { if (lead) ld1(gR + s * X_KS + (cg * 2 + lw) * 1024, lds0 + LDS_SCR + lw * 1024, voffA); };
        wait_vm<12>();
        unsigned sn = 0;
        for (int s = 1; s < KSTEPS - 5; ++s) {
            bar();
            early(s + 3);
            if (s + 2 != pk) {
#pragma unroll
                for (int i = 0; i < 4; ++i) { ldB(s + 2, sn, i); ldA(s + 2, sn, i); }
            } else {
#pragma unroll
                for (int i = 0; i < 4; ++i) {
                    unsigned keep;
                    asm volatile("s_mov_b32 %0, m0\n\ts_mov_b32 m0, %3\n\ts_nop 0\n\tglobal_load_lds_dwordx4 %1, %2 sc0 sc1\n\ts_mov_b32 m0, %0"
                                 : "=&s"(keep) : "v"(voffA), "s"(gP + lw * 4096 + i * 1024), "s"(ldsA + sn + i * 1024) : "memory");
                    ldB(s + 2, sn, i);
                }
            }
            sn = (sn == (D - 1) * STG) ? 0u : sn + STG;
            if (lead) wait_vm<13>(); else wait_vm<12>();
        }
        const unsigned hoff = (lw >> 1) * 8192 + (lw & 1) * 2048;
        auto ldH = [&](int s, int half) INL {
            ld1(gR + s * X_KS + hoff + half * 4096, lds0 + (s % D) * STG + hoff + half * 4096, voffA);
            ld1(gR + s * X_KS + hoff + half * 4096 + 1024, lds0 + (s % D) * STG + hoff + half * 4096 + 1024, voffA);
        };
        auto ldU = [&](int s) INL {
            ldH(s, 0);
#pragma unroll
            for (int i = 0; i < 4; ++i) ldB(s, (s % D) * STG, i);
        };
        bar();
        early(30); ldU(29);
        if (lead) wait_vm<11>(); else wait_vm<10>();
        bar();
        early(31); ldU(30);
        if (lead) wait_vm<11>(); else wait_vm<10>();
        bar();
        ldU(31);
        wait_vm<10>();
        bar();
        ldH(29, 1); ldH(30, 1); ldH(31, 1);
        wait_vm<6>();
        bar();
        wait_vm<4>();
        bar();
        wait_vm<2>();
        bar();
        wait_vm<0>();
        bar();
    }
}

}

extern "C" void kernel_launch(int rank, int nranks, ncclComm_t comm,
                              hipStream_t stream,
                              void* const* d_in, const int* in_sizes, int n_in,
                              void* d_out, int out_size,
                              const int* in_dtypes, int out_dtype,
                              void* const* d_out_all,
                              void* const* d_ws_all, size_t ws_bytes) {
    if (nranks != NR || (unsigned)rank >= (unsigned)NR || n_in != 4 || ws_bytes < WS_NEED ||
        in_sizes[0] != MS * KD || in_sizes[1] != KD * ND || out_size != NR * MS * NS) {
        fprintf(stderr, "kernel_launch: unexpected shapes\n");
        return;
    }
    Args a;
    a.x = (const signed char*)d_in[0]; a.w = (const signed char*)d_in[1];
    a.sx = (const float*)d_in[2]; a.sw = (const float*)d_in[3];
    a.out = (float*)d_out; a.rank = rank; a.myws = (char*)d_ws_all[rank]; a.stamps = nullptr;
    static const unsigned kXmap[NR] = {0x74513206u, 0x26305417u, 0x17546023u, 0x47150632u,
                                       0x36027145u, 0x60231754u, 0x51472360u, 0x02364571u};
    a.xmap = kXmap[rank];
    for (int p = 0; p < NR; ++p) a.ws_all[p] = (char*)d_ws_all[p];
    hipLaunchKernelGGL(gemm_gather_kernel, dim3(GRID), dim3(BLOCK), 0, stream, a);
    HIP_CHECK(hipGetLastError());
}
```

```cpp
#include <hip/hip_runtime.h>
#include <rccl/rccl.h>
#include <cstdio>

#define HIP_CHECK(expr) do { hipError_t _e = (expr); if (_e != hipSuccess) { \
    fprintf(stderr, "HIP error %s at %s:%d\n", hipGetErrorString(_e), __FILE__, __LINE__); \
    return; } } while (0)

namespace {

#define LAS __attribute__((address_space(3)))
typedef unsigned u32x4 __attribute__((ext_vector_type(4)));
typedef int i32x4 __attribute__((ext_vector_type(4)));
typedef int i32x2 __attribute__((ext_vector_type(2)));

constexpr int NR = 8;
constexpr int KD = 4096, ND = 16384;
constexpr int MS = 512, NS = 2048;
constexpr int GRID = 256, BLOCK = 768;
constexpr int IMG = 16384;
constexpr int STG = 3 * IMG;
constexpr int KSTEPS = KD / 128;
constexpr int D = 3;
constexpr int LDS_CTL = D * STG;
constexpr int LDS_SCR = LDS_CTL + 1024;
constexpr int LDS_BYTES = LDS_SCR + 2048;
constexpr int NFIRST = 6;
constexpr int NP = 2, PK0 = 5;

constexpr size_t WS_CNT = 0;
constexpr size_t WS_TMO = 4096;
constexpr size_t WS_FLAG = 8192;
constexpr size_t WS_CLAIM = 32768;
constexpr size_t WS_PAD = 65536;
constexpr size_t WS_X = 131072;
constexpr size_t X_KS = 8 * IMG + 256, X_PAR = 4 * IMG, X_RG = IMG;
constexpr size_t WS_R = WS_X + KSTEPS * X_KS;
constexpr size_t WS_NEED = WS_R + (size_t)2 * NR * NP * IMG;
static_assert(WS_FLAG + (size_t)NR * 8 * 128 <= WS_CLAIM && WS_CLAIM + (size_t)GRID * 128 <= WS_PAD, "layout");
constexpr unsigned long long DEADLINE_TICKS = 50000000ull;

struct Args {
    const signed char* x; char* myws; const float* sx; const float* sw;
    const signed char* w; float* out; int rank; unsigned xmap;
    unsigned long long pad;
    char* ws_all[NR];
};

template <int N> __device__ __forceinline__ void wait_vm() { asm volatile("s_waitcnt vmcnt(%0)" :: "n"(N) : "memory"); }
__device__ __forceinline__ void bar() { asm volatile("" ::: "memory"); __builtin_amdgcn_s_barrier(); asm volatile("" ::: "memory"); }
#define SB0 __builtin_amdgcn_sched_barrier(0)
#define INL __attribute__((always_inline))

__device__ __forceinline__ bool wait_count(const char* f, int lane, unsigned want, unsigned call, unsigned* tmo) {
    const unsigned long long t0 = __builtin_amdgcn_s_memrealtime();
    for (unsigned it = 1;; ++it) {
        unsigned v = 0;
        if (lane == 0) asm volatile("global_atomic_add %0, %1, %1, %2 sc0 sc1\n\ts_waitcnt vmcnt(0)" : "=&v"(v) : "v"(0), "s"(f) : "memory");
        if (__builtin_amdgcn_readfirstlane(v) >= want) return false;
        if ((it & 63u) == 0u) {
            if (__builtin_amdgcn_readfirstlane(__hip_atomic_load(tmo, __ATOMIC_RELAXED, __HIP_MEMORY_SCOPE_AGENT)) == call
                || __builtin_amdgcn_s_memrealtime() - t0 > DEADLINE_TICKS) {
                __hip_atomic_store(tmo, call, __ATOMIC_RELAXED, __HIP_MEMORY_SCOPE_AGENT);
                return true;
            }
        }
        if (it > 64u) __builtin_amdgcn_s_sleep(4);
    }
}

static_assert(PK0 >= 3 && PK0 + 7 * NP <= 29 && NP == 2, "pushed images belong to the main loop, one per row group and peer");

struct Frags { i32x4 a[4], b[4]; };

__global__ __launch_bounds__(BLOCK, 3) void gemm_gather_kernel(Args a) {
    __shared__ __attribute__((aligned(16))) char smem[LDS_BYTES];
    LAS char* lds = (LAS char*)smem;
    const int tid = threadIdx.x, lane = tid & 63, wid = __builtin_amdgcn_readfirstlane(tid >> 6);
    const int b = blockIdx.x;
    unsigned kw0, kw1;
    asm volatile("s_load_dword %0, %2, 0x0\n\ts_load_dword %1, %2, 0x40" : "=&s"(kw0), "=&s"(kw1) : "s"(__builtin_amdgcn_kernarg_segment_ptr()));
    volatile LAS unsigned* ctl = (volatile LAS unsigned*)(lds + LDS_CTL);
#define KARGS asm volatile("" :: "s"(a.x), "s"(a.myws), "s"(a.w), "s"(a.rank), "s"(a.xmap), "s"(a.ws_all[0]), "s"(a.ws_all[1]), "s"(a.ws_all[2]), \
                 "s"(a.ws_all[3]), "s"(a.ws_all[4]), "s"(a.ws_all[5]), "s"(a.ws_all[6]), "s"(a.ws_all[7])); \
              asm volatile("s_waitcnt lgkmcnt(0)" :: "s"(kw0), "s"(kw1)); SB0
#define KLOCALS const int rank = a.rank; char* myws = a.myws; unsigned* tmo = (unsigned*)(myws + WS_TMO)

    if (__builtin_expect(wid < 8, 1)) {
        __builtin_amdgcn_s_setprio(3);
        const int prg = (b >> 1) & 3, pks = b >> 3, p = (b & 1) * 8192 + tid * 16;
        const int pj = pks - PK0;
        const bool push = pj >= 0 && pj < 7 * NP && prg == 1 + pj % 3;
        unsigned xo = (prg * 128 + (p >> 7)) * KD + pks * 128 + ((((p >> 4) & 7) ^ ((p >> 8) & 7)) << 4), co = WS_CNT + b * 4;
        const int late = __builtin_amdgcn_readfirstlane(pks >= NFIRST / 2 && !(push && pj < 7));
        asm volatile("" : "+v"(xo), "+v"(co) : "s"(late));
        SB0;
        KARGS;
        KLOCALS;
        unsigned call;
        {
            u32x4 v;
            asm volatile("s_cmp_eq_u32 %6, 0\n\ts_cbranch_scc1 1f\n\ts_sleep 8\n1:\n\t"
                         "global_load_dwordx4 %0, %2, %4\n\tglobal_load_dword %1, %3, %5 sc1"
                         : "=&v"(v), "=&v"(call) : "v"(xo), "v"(co), "s"(a.x), "s"(myws), "s"(late) : "memory", "scc");
            unsigned wlo = 0, whi = 0;
#pragma unroll
            for (int q = 0; q < NR; ++q)
                asm volatile("v_writelane_b32 %0, %2, %4\n\tv_writelane_b32 %1, %3, %4" : "+v"(wlo), "+v"(whi)
                             : "s"((unsigned)(size_t)a.ws_all[q]), "s"((unsigned)((size_t)a.ws_all[q] >> 32)), "n"(q));
            char* flag = (char*)(((size_t)whi << 32) | wlo) + WS_FLAG + (size_t)((rank * 4 + prg) * 2 + (pks >= NFIRST / 2)) * 128;
            const char* dst = myws + WS_X + prg * X_RG + pks * X_KS;
            const char* dst2 = a.ws_all[(rank + 1 + (push ? pj % 7 : 0)) & 7] + WS_R + (size_t)(rank * NP + (pj >= 7)) * IMG;
            unsigned cadr = (unsigned)(size_t)lds + LDS_CTL + 8, one = 1u, arrived;
            asm volatile("" : "+v"(flag), "+v"(cadr), "+v"(one) : "s"(dst), "s"(dst2));
            bar();
            asm volatile("s_waitcnt vmcnt(0)" : "+v"(v), "+v"(call) :: "memory");
            const unsigned odd = ~call & 1u;
            asm volatile("global_store_dwordx4 %0, %1, %2 sc0 sc1\n\ts_nop 1" :: "v"(p + odd * (unsigned)X_PAR), "v"(v), "s"(dst) : "memory");
            if (push) {
                if (pj >= 7) wait_vm<0>();
                asm volatile("global_store_dwordx4 %0, %1, %2 sc0 sc1\n\ts_nop 1" :: "v"(p + odd * (unsigned)(NR * NP * IMG)), "v"(v), "s"(dst2) : "memory");
            }
            call = __builtin_amdgcn_readfirstlane(call) + 1u;
            asm volatile("s_waitcnt vmcnt(0)\n\ts_mov_b64 exec, 1\n\tds_add_rtn_u32 %0, %1, %2\n\ts_mov_b64 exec, -1\n\ts_waitcnt lgkmcnt(0)"
                         : "=&v"(arrived) : "v"(cadr), "v"(one) : "memory");
            if (__builtin_amdgcn_readfirstlane(arrived) == 7u)
                asm volatile("s_mov_b64 exec, 0xff\n\tglobal_atomic_add %0, %1, off sc1\n\ts_mov_b64 exec, -1" :: "v"(flag), "v"(one) : "memory");
        }
        __builtin_amdgcn_s_setprio(0);
        const int wm = wid >> 2, wn = wid & 3;
        const int q8 = (lane & 15) >> 1;
        unsigned aoff[2], boff[4];
        aoff[0] = (wm * 64 + (lane & 15)) * 128 + (((lane >> 4) ^ q8) << 4); aoff[1] = aoff[0] ^ 64u;
#pragma unroll
        for (int j = 0; j < 4; ++j)
            boff[j] = IMG + ((lane >> 4) * 16 + q8) * 256 + (((wn * 4 + j) ^ (q8 | (((lane >> 4) & 1) << 3))) << 4) + (lane & 1) * 8;
        i32x4 acc[4][4];
#pragma unroll
        for (int i = 0; i < 4; ++i)
#pragma unroll
            for (int j = 0; j < 4; ++j) acc[i][j] = (i32x4){0, 0, 0, 0};

        auto rdB1 = [&](Frags& f, unsigned so, int h, int j) INL {
            const unsigned ba = boff[j] + so + h * 16384;
            const i32x2 lo = __builtin_amdgcn_ds_read_tr8_b64_v2i32((LAS i32x2*)(lds + ba));
            const i32x2 hi = __builtin_amdgcn_ds_read_tr8_b64_v2i32((LAS i32x2*)(lds + ba + 2048));
            f.b[j] = (i32x4){lo.x, lo.y, hi.x, hi.y};
        };
        auto rdA1 = [&](Frags& f, unsigned so, int h, int i) INL { f.a[i] = *(const LAS i32x4*)(lds + aoff[h] + so + i * 2048); };
        auto rd = [&](Frags& f, unsigned so, int h) INL {
#pragma unroll
            for (int t = 0; t < 4; ++t) { rdB1(f, so, h, t); rdA1(f, so, h, t); }
        };
        auto rdB = [&](Frags& f, unsigned so, int h) INL {
#pragma unroll
            for (int t = 0; t < 4; ++t) rdB1(f, so, h, t);
        };
        auto rdA = [&](Frags& f, unsigned so, int h, int half) INL { rdA1(f, so, h, 2 * half); rdA1(f, so, h, 2 * half + 1); };
        auto mmr = [&](const Frags& f, int i) INL {
#pragma unroll
            for (int j = 0; j < 4; ++j) acc[i][j] = __builtin_amdgcn_mfma_i32_16x16x64_i8(f.a[i], f.b[j], acc[i][j], 0, 0, 0);
        };
        auto mmh = [&](const Frags& f, int half) INL { mmr(f, 2 * half); mmr(f, 2 * half + 1); };
        auto mm = [&](const Frags& f) INL { mmh(f, 0); mmh(f, 1); };
        auto sched = [&]() INL {
#pragma unroll
            for (int i = 0; i < 12; ++i) { __builtin_amdgcn_sched_group_barrier(0x100, 1, 0); __builtin_amdgcn_sched_group_barrier(0x8, 1, 0); }
            __builtin_amdgcn_sched_group_barrier(0x8, 4, 0);
        };
        bar();
        const unsigned slot = __builtin_amdgcn_readfirstlane(ctl[1]);
        const int peer = (a.xmap >> (4 * (slot >> 5))) & 7, rg = slot & 3, cg = (slot >> 2) & 7;
        const char* ob = (const char*)(a.out + (size_t)(peer * MS + rg * 128 + wm * 64) * NS + cg * 256 + wn * 64);
        float s0, s1;
        asm volatile("s_load_dword %0, %2, 0x0\n\ts_load_dword %1, %3, 0x0\n\ts_waitcnt lgkmcnt(0)"
                     : "=&s"(s0), "=&s"(s1) : "s"(a.sx), "s"(a.sw) : "memory");
        float scale = s0 * s1;
        if (wid == 2)
            asm volatile("s_load_dword %0, %2, 0x0\n\ts_load_dword %1, %3, 0x0\n\ts_waitcnt lgkmcnt(0)"
                         : "=&s"(s0), "=&s"(s1) : "s"((const char*)a.w + ((size_t)(slot & 31) << 21)), "s"(ob + (size_t)(cg & 1) * 127 * NS * 4) : "memory");
        if (wid == 1) {
            __builtin_amdgcn_s_sleep(20);
            if (wait_count(myws + WS_FLAG + ((peer * 4 + rg) * 2 + 1) * 128, lane, call * (64 - NFIRST), call, tmo)) ctl[0] = 1u;
            asm volatile("s_waitcnt vmcnt(0) lgkmcnt(0)" ::: "memory");
        }
        Frags f0, f1;
        unsigned sb = 0;
        bar();
        if (tid == 0) __hip_atomic_store((unsigned*)(myws + WS_CNT) + b, call, __ATOMIC_RELAXED, __HIP_MEMORY_SCOPE_AGENT);
        rd(f0, sb, 0);
        for (int s = 0; s < KSTEPS - 4; ++s) {
            SB0; rd(f1, sb, 1); mm(f0); sched(); SB0;
            asm volatile("s_waitcnt lgkmcnt(0)" ::: "memory");
            bar();
            sb = (sb == (D - 1) * STG) ? 0u : sb + STG;
            rd(f0, sb, 0); mm(f1); sched();
        }
        unsigned S28 = (28 % D) * STG, S29 = (29 % D) * STG, S30 = (30 % D) * STG, S31 = (31 % D) * STG;
        asm volatile("" : "+s"(S28), "+s"(S29), "+s"(S30), "+s"(S31));
        if (__builtin_amdgcn_readfirstlane(ctl[0]) != 0u) scale = __builtin_nanf("");
        asm volatile("" : "+v"(scale));
        SB0; rd(f1, S28, 1); mm(f0); sched(); SB0;
        asm volatile("s_waitcnt lgkmcnt(0)" ::: "memory");
        bar();
        rdA(f0, S29, 0, 0); rdB(f0, S29, 0); SB0; mm(f1); SB0;
        rdA(f1, S29, 1, 0); rdB(f1, S29, 1); SB0; mmh(f0, 0); SB0;
        bar();
        rdA(f0, S30, 0, 0); rdB(f0, S30, 0); SB0; mmh(f1, 0); SB0;
        rdA(f1, S30, 1, 0); rdB(f1, S30, 1); SB0; mmh(f0, 0); SB0;
        bar();
        rdA(f0, S31, 0, 0); rdB(f0, S31, 0); SB0; mmh(f1, 0); SB0;
        rdA(f1, S31, 1, 0); rdB(f1, S31, 1); SB0; mmh(f0, 0); SB0;
        rdB(f0, S29, 0); SB0; mmh(f1, 0); SB0;
        rdB(f1, S29, 1); SB0;
        unsigned ov;
        asm volatile("v_mbcnt_lo_u32_b32 %0, -1, 0\n\tv_mbcnt_hi_u32_b32 %0, -1, %0" : "=v"(ov));
        ov = (ov >> 5) * 8 * NS * 4 + (ov & 31) * 4;
        auto st = [&](int n) INL {
            const int i = n >> 3, jp = (n >> 2) & 1, e = n & 3;
            const auto sw = __builtin_amdgcn_permlane16_swap((unsigned)acc[i][2 * jp][e], (unsigned)acc[i][2 * jp + 1][e], false, false);
            const float y0 = __builtin_elementwise_maximum((float)(int)sw[0] * scale, 0.0f);
            const float y1 = __builtin_elementwise_maximum((float)(int)sw[1] * scale, 0.0f);
            const char* rb = ob + (size_t)(i * 16 + e) * NS * 4 + jp * 128;
            asm volatile("global_store_dword %0, %1, %3 sc1\n\tglobal_store_dword %0, %2, %4 sc1"
                         :: "v"(ov), "v"(y0), "v"(y1), "s"(rb), "s"(rb + 4 * NS * 4) : "memory");
        };
#pragma unroll
        for (int n = 0; n < 9; ++n) { st(n); SB0; }
        bar();
        rdA(f0, S29, 0, 1); rdA(f1, S29, 1, 1); SB0; mmh(f0, 1); SB0; rdB(f0, S30, 0); SB0; mmh(f1, 1); SB0; rdB(f1, S30, 1); SB0;
#pragma unroll
        for (int n = 9; n < 13; ++n) { st(n); SB0; }
        bar();
        rdA(f0, S30, 0, 1); rdA(f1, S30, 1, 1); SB0; mmh(f0, 1); SB0; rdB(f0, S31, 0); SB0; mmh(f1, 1); SB0; rdB(f1, S31, 1); SB0;
#pragma unroll
        for (int n = 13; n < 16; ++n) { st(n); SB0; }
        bar();
        rdA(f0, S31, 0, 1); rdA(f1, S31, 1, 1); SB0; mmh(f0, 1); mmh(f1, 1); SB0;
#pragma unroll
        for (int n = 16; n < 32; ++n) { st(n); SB0; }
    } else {
        KARGS;
        KLOCALS;
        const int lw = wid - 8;
        const unsigned xcc = (unsigned)__builtin_amdgcn_s_getreg(20 | (3 << 11)) & 7u;
        unsigned* claim = (unsigned*)(myws + WS_CLAIM);
        unsigned slot = xcc * 32u + (unsigned)(b >> 3);
        unsigned call, old = 0;
        asm volatile("global_load_dword %0, %1, %2 sc1" : "=v"(call) : "v"((unsigned)WS_CNT + b * 4), "s"(myws) : "memory");
        if (lw == 0 && lane == 0)
            asm volatile("global_atomic_add %0, %1, %2, %3 sc0" : "+v"(old) : "v"(0), "v"(1), "s"(claim + slot * 32) : "memory");
        unsigned touch = 0;
        if (lw == 3 && (b >> 3) >= 3 && (b >> 3) < 5) {
            const char* pw = a.ws_all[(a.xmap >> (4 * xcc)) & 7];
            const char* t = lane < 12 ? myws : lane < 16 ? pw : (const char*)a.w;
            unsigned lo = (unsigned)(size_t)t, hi = (unsigned)((size_t)t >> 32);
#pragma unroll
            for (int q = 0; q < NR; ++q)
                asm volatile("v_writelane_b32 %0, %2, %4\n\tv_writelane_b32 %1, %3, %4" : "+v"(lo), "+v"(hi)
                             : "s"((unsigned)(size_t)a.ws_all[q]), "s"((unsigned)((size_t)a.ws_all[q] >> 32)), "n"(q));
            const unsigned o0 = WS_PAD + (rank * 32 + (b & 31)) * 128, o1 = WS_X + ((lane & 3) * 8 + 8) * X_KS - 128, o2 = (lane & 3) << 21;
            t = (const char*)(((size_t)hi << 32) | lo) + (lane < 8 ? o0 : lane < 16 ? o1 : o2);
            if (lane < 8 || lane == 16) asm volatile("global_load_dword %0, %1, off sc0 sc1" : "+v"(touch) : "v"(t) : "memory");
            wait_vm<1>();
            if (lane >= 8 && lane < 19 && lane != 16) asm volatile("global_load_dword %0, %1, off sc0 sc1" : "+v"(touch) : "v"(t) : "memory");
        }
        const unsigned lds0 = (unsigned)(size_t)lds, ldsA = lds0 + lw * 4096, ldsB = lds0 + IMG + lw * 8192;
        const unsigned voffA = lane * 16;
        unsigned voff[4];
#pragma unroll
        for (int i = 0; i < 4; ++i) { const int k = (i & 1) * 4 + (lane >> 4);
            voff[i] = k * ND + (((lane & 15) ^ (k | ((i >> 1) << 3))) << 4); }
        const char* gW = (const char*)a.w + (size_t)rank * NS + (size_t)(lw * 32) * ND;
        auto ld1 = [&](const char* g, unsigned l, unsigned v) INL {
            unsigned keep;
            asm volatile("s_mov_b32 %0, m0\n\ts_mov_b32 m0, %3\n\ts_nop 0\n\tglobal_load_lds_dwordx4 %1, %2\n\ts_mov_b32 m0, %0"
                         : "=&s"(keep) : "v"(v), "s"(g), "s"(l) : "memory");
        };
        if (lw == 0 && lane < 4) ctl[lane] = 0u;
        asm volatile("s_waitcnt lgkmcnt(0)" ::: "memory");
        bar();
        asm volatile("s_waitcnt vmcnt(0)" : "+v"(call), "+v"(old), "+v"(touch) :: "memory");
        call = __builtin_amdgcn_readfirstlane(call) + 1u;
        if (lw == 0) {
            unsigned lost = 0;
            if (lane == 0) {
#pragma nounroll
                for (unsigned tries = 0; old != call - 1u; ++tries) {
                    __hip_atomic_fetch_sub(claim + slot * 32, 1u, __ATOMIC_RELAXED, __HIP_MEMORY_SCOPE_AGENT);
                    if (tries >= GRID) { lost = 1u; break; }
                    slot = (slot + 1u) & (GRID - 1);
                    old = __hip_atomic_fetch_add(claim + slot * 32, 1u, __ATOMIC_RELAXED, __HIP_MEMORY_SCOPE_AGENT);
                }
            }
            slot = __builtin_amdgcn_readfirstlane(slot);
            lost = __builtin_amdgcn_readfirstlane(lost);
            const int pr = (a.xmap >> (4 * (slot >> 5))) & 7;
            const char* g0 = a.ws_all[pr] + WS_X + (call & 1u) * X_PAR + (slot & 3) * X_RG;
            asm volatile("" :: "s"(g0));
            ctl[1] = slot;
            const char* fp = myws + WS_FLAG + (pr * 4 + (slot & 3)) * 2 * 128;
            unsigned left = lost ? 0u : 64u;
            for (int pass = 0; pass < 2; ++pass) {
                unsigned tv, ts_, keep;
                asm volatile("s_mov_b32 %3, m0\n\ts_mov_b64 exec, 1\n"
                             "1:\n\ts_sub_u32 %0, %0, 1\n\ts_cbranch_scc1 2f\n\t"
                             "global_atomic_add %1, %4, %4, %5 sc0 sc1\n\ts_waitcnt vmcnt(0)\n\tv_readfirstlane_b32 %2, %1\n\ts_nop 0\n\t"
                             "s_cmp_lt_u32 %2, %6\n\ts_cbranch_scc1 1b\n\t"
                             "s_mov_b64 exec, -1\n\ts_mov_b32 m0, %12\n\ts_nop 0\n\t"
                             "global_load_lds_dwordx4 %7, %8\n\tglobal_load_lds_dwordx4 %7, %8 offset:1024\n\ts_add_u32 m0, m0, 0x1000\n\ts_nop 0\n\t"
                             "global_load_lds_dwordx4 %7, %9\n\tglobal_load_lds_dwordx4 %7, %9 offset:1024\n\ts_add_u32 m0, m0, 0x1000\n\ts_nop 0\n\t"
                             "global_load_lds_dwordx4 %7, %10\n\tglobal_load_lds_dwordx4 %7, %10 offset:1024\n\ts_add_u32 m0, m0, 0x1000\n\ts_nop 0\n\t"
                             "global_load_lds_dwordx4 %7, %11\n\tglobal_load_lds_dwordx4 %7, %11 offset:1024\n\ts_mov_b32 %0, 0\n"
                             "2:\n\ts_mov_b64 exec, -1\n\ts_mov_b32 m0, %3"
                             : "+&s"(left), "=&v"(tv), "=&s"(ts_), "=&s"(keep)
                             : "v"(0), "s"(fp), "s"(call * NFIRST), "v"(voffA), "s"(g0), "s"(g0 + 4096), "s"(g0 + 8192), "s"(g0 + 12288), "s"(lds0)
                             : "memory", "scc");
                if (__builtin_amdgcn_readfirstlane(left) == 0u) break;
                if (lost || wait_count(fp, lane, call * NFIRST, call, tmo)) { ctl[0] = 1u; break; }
                left = 1u;
            }
            asm volatile("s_waitcnt lgkmcnt(0)" ::: "memory");
        }
        bar();
        const bool bad = __builtin_amdgcn_readfirstlane(ctl[0]) != 0u;
        slot = __builtin_amdgcn_readfirstlane(ctl[1]);
        const int peer = (a.xmap >> (4 * (slot >> 5))) & 7, rg = slot & 3, cg = (slot >> 2) & 7;
        const char* gB = gW + cg * 256;
        const char* gR = a.ws_all[bad ? rank : peer] + WS_X + (call & 1u) * X_PAR + rg * X_RG;
        auto ldA = [&](int s, unsigned so, int i) INL { ld1(gR + s * X_KS + lw * 4096 + i * 1024, ldsA + so + i * 1024, voffA); };
        auto ldB = [&](int s, unsigned so, int i) INL {
            const char* g = gB + (size_t)(s * 128 + i * 8) * ND;
            ld1(g, ldsB + so + i * 2048, voff[i & 2]);
            ld1(g, ldsB + so + i * 2048 + 1024, voff[(i & 2) + 1]);
        };
        auto A = [&](int n) INL { ldA(n >> 2, (n >> 2) * STG, n & 3); };
        auto B = [&](int n) INL { ldB(n >> 2, (n >> 2) * STG, n & 3); };
        B(0); A(2); B(1); A(3); B(2); A(4); B(3);
        A(5); B(4); A(6); B(5); B(6); A(7); B(7);
        A(8); B(8); A(9); B(9); B(10); A(10); B(11); A(11);
        wait_vm<23>();
        bar();
        if (__builtin_amdgcn_readfirstlane(ctl[0]) != 0u) gR = myws + WS_X;
        int pk = -1; const char* gP = gR;
#pragma unroll
        for (int n = 0; n < NP; ++n) {
            const int j = ((rank - peer - 1) & 7) + 7 * n;
            if (peer != rank && rg == 1 + j % 3) { pk = PK0 + j; gP = myws + WS_R + (size_t)(((call & 1u) * NR + peer) * NP + n) * IMG; }
        }
        const bool lead = peer != rank && rg == 0 && lw < 2;
        auto early = [&](int s) INL { if (lead) ld1(gR + s * X_KS + (cg * 2 + lw) * 1024, lds0 + LDS_SCR + lw * 1024, voffA); };
        wait_vm<12>();
        unsigned sn = 0;
        for (int s = 1; s < KSTEPS - 5; ++s) {
            bar();
            early(s + 3);
            if (s + 2 != pk) {
#pragma unroll
                for (int i = 0; i < 4; ++i) { ldB(s + 2, sn, i); ldA(s + 2, sn, i); }
            } else {
#pragma unroll
                for (int i = 0; i < 4; ++i) {
                    unsigned keep;
                    asm volatile("s_mov_b32 %0, m0\n\ts_mov_b32 m0, %3\n\ts_nop 0\n\tglobal_load_lds_dwordx4 %1, %2 sc0 sc1\n\ts_mov_b32 m0, %0"
                                 : "=&s"(keep) : "v"(voffA), "s"(gP + lw * 4096 + i * 1024), "s"(ldsA + sn + i * 1024) : "memory");
                    ldB(s + 2, sn, i);
                }
            }
            sn = (sn == (D - 1) * STG) ? 0u : sn + STG;
            if (lead) wait_vm<13>(); else wait_vm<12>();
        }
        const unsigned hoff = (lw >> 1) * 8192 + (lw & 1) * 2048;
        auto ldH = [&](int s, int half) INL {
            ld1(gR + s * X_KS + hoff + half * 4096, lds0 + (s % D) * STG + hoff + half * 4096, voffA);
            ld1(gR + s * X_KS + hoff + half * 4096 + 1024, lds0 + (s % D) * STG + hoff + half * 4096 + 1024, voffA);
        };
        auto ldU = [&](int s) INL {
            ldH(s, 0);
#pragma unroll
            for (int i = 0; i < 4; ++i) ldB(s, (s % D) * STG, i);
        };
        bar();
        early(30); ldU(29);
        if (lead) wait_vm<11>(); else wait_vm<10>();
        bar();
        early(31); ldU(30);
        if (lead) wait_vm<11>(); else wait_vm<10>();
        bar();
        ldU(31);
        wait_vm<10>();
        bar();
        ldH(29, 1); ldH(30, 1); ldH(31, 1);
        wait_vm<6>();
        bar();
        wait_vm<4>();
        bar();
        wait_vm<2>();
        bar();
        wait_vm<0>();
        bar();
    }
}

}


extern "C" void kernel_launch(int rank, int nranks, ncclComm_t comm,
                              hipStream_t stream,
                              void* const* d_in, const int* in_sizes, int n_in,
                              void* d_out, int out_size,
                              const int* in_dtypes, int out_dtype,
                              void* const* d_out_all,
                              void* const* d_ws_all, size_t ws_bytes) {
    if (nranks != NR || (unsigned)rank >= (unsigned)NR || n_in != 4 || ws_bytes < WS_NEED ||
        in_sizes[0] != MS * KD || in_sizes[1] != KD * ND || out_size != NR * MS * NS) {
        fprintf(stderr, "kernel_launch: unexpected shapes\n");
        return;
    }
    Args a;
    a.x = (const signed char*)d_in[0]; a.w = (const signed char*)d_in[1];
    a.sx = (const float*)d_in[2]; a.sw = (const float*)d_in[3];
    a.out = (float*)d_out; a.rank = rank; a.myws = (char*)d_ws_all[rank]; a.pad = 0;
    constexpr unsigned kXmap[NR] = {0x74513206u, 0x26305417u, 0x17546023u, 0x47150632u,
                                    0x36027145u, 0x60231754u, 0x51472360u, 0x02364571u};
    a.xmap = kXmap[rank];
    for (int p = 0; p < NR; ++p) a.ws_all[p] = (char*)d_ws_all[p];
    hipLaunchKernelGGL(gemm_gather_kernel, dim3(GRID), dim3(BLOCK), 0, stream, a);
    HIP_CHECK(hipGetLastError());
}
```
